```python
import numpy as np

kernel_name = "gpu_a2a_gemm_m4096_k4096_n16384_f32_none"


ABSMAX_RTOL = 0.001

COMPUTE_DTYPE = "float32"
EPILOGUE = "none"
IN_QUANT = None
OUT_QUANT = None
IN_DTYPES = {"x": "float32", "w_mat": "float32"}
WORLD_SIZE = 8

def setup_inputs():

    x = np.random.randn(4096, 4096).astype(np.float32)
    w_mat = (np.random.randn(4096, 16384) / np.sqrt(4096)).astype(np.float32)
    return {"x": x, "w_mat": w_mat}

def _epilogue(y):
    return y

def reference(x, w_mat):
    y = (x @ w_mat).astype(np.float32)
    return _epilogue(y)
```

```cpp
#include <hip/hip_runtime.h>
#include <rccl/rccl.h>
#include <cstdio>

#define HIP_CHECK(expr) do { hipError_t _e = (expr); if (_e != hipSuccess) { \
    fprintf(stderr, "HIP error %s at %s:%d\n", hipGetErrorString(_e), __FILE__, __LINE__); \
    return; } } while (0)

#define KDBG 0
#define LFB 1
#define LFS 1
#define SKEW 0
#define PMAP 1
typedef unsigned u32x4 __attribute__((ext_vector_type(4)));
namespace {
constexpr int N_ = 16384, NR = 8;
constexpr unsigned WS_A = 0;
constexpr unsigned WS_FLAG = 8u << 20;
constexpr unsigned WS_CNT = WS_FLAG + 65536;
constexpr unsigned WS_TMO = WS_FLAG + 131072;
constexpr unsigned WS_DBG = WS_FLAG + 262144;
constexpr unsigned WS_USED = 16u << 20;
}
#define R10(p, a) p #a "0", p #a "1", p #a "2", p #a "3", p #a "4", p #a "5", p #a "6", p #a "7", p #a "8", p #a "9"
#define VCLOB "v0","v1","v2","v3","v4","v5","v6","v7","v8","v9", R10("v",1),R10("v",2),R10("v",3),R10("v",4),R10("v",5),R10("v",6),R10("v",7),R10("v",8),R10("v",9), \
  R10("v",10),R10("v",11),R10("v",12),R10("v",13),R10("v",14),R10("v",15),R10("v",16),R10("v",17),R10("v",18),R10("v",19),R10("v",20),R10("v",21),R10("v",22),R10("v",23), \
  "v240","v241","v242","v243","v244","v245","v246","v247","v254","v255"
#define RFL(x) __builtin_amdgcn_readfirstlane(x)
#define SCLOB R10("s",7), R10("s",8), "s90", "s91", "s92", "s93", "s94", "s95", "s96", "s97"

__device__ __forceinline__ u32x4 make_srd(const void* p, unsigned bytes) {
  u32x4 r = {(unsigned)(size_t)p, (unsigned)((size_t)p >> 32), bytes, 0x00020000u};
  return r;
}

__global__ __launch_bounds__(512, 2) void a2a_gemm_kernel(const float* __restrict__ x, const float* __restrict__ W, float* C,
                                                          void* const* __restrict__ ws_all, int rank) {
  __shared__ __attribute__((aligned(16))) char lds[163840];
  const int wave = __builtin_amdgcn_readfirstlane(threadIdx.x >> 6);
  const int lane = threadIdx.x & 63;
  const int b = blockIdx.x;
  const int xcd = b & 7, j = b >> 3;
  const int mh = j & 1, slab = PMAP ? (j >> 1) * 8 + xcd : xcd * 16 + (j >> 1);
  const int n0 = slab * 128;
  const unsigned ldsb = (unsigned)(size_t)(__attribute__((address_space(3))) char*)lds;
  const int grp = wave >> 2, w = wave & 3;
  const unsigned dest = (unsigned)(rank + (j & 7)) & 7u;
  const int kt = (j >> 3) * 2 + grp;
  const int m16s = xcd * 4 + w;
  const unsigned self = (j & 7) == 0;
  char* wsl = (char*)ws_all[(unsigned)rank & 7u];
  char* wsd = (char*)ws_all[dest];
  const u32x4 srdw = make_srd(W, 256u << 20), srdx = make_srd(x, 8u << 20), srdc = make_srd(C, 32u << 20);
  const u32x4 srds = make_srd(wsl, WS_USED), srdd = make_srd(wsd, WS_USED);
  const unsigned v16 = lane * 16;
  const unsigned vw = (unsigned)(((32 * (w >> 1) + 4 * (lane >> 4)) * N_ + n0 + 64 * (w & 1) + 4 * (lane & 15)) * 4);
  unsigned vx = (unsigned)(((dest * 512 + m16s * 16 + (lane & 15)) * 512 + kt * 64 + 4 * (lane >> 4)) * 4);
  unsigned tk, vp = 0;
  {
    const int ty = LFS == 1 ? (j >> 1) & 1 : LFS == 2 ? (xcd >> 1) & 1 : 0;
    int q = lane; const int z = lane - ty * LFB;
    if (LFB && mh && z >= 0 && z < 64 - 2 * LFB * ty && !(ty && lane >= 8 - LFB && lane < 8 + LFB)) q = (z ^ LFB) + ty * LFB;
    const int u = q - 8;
    const int i = 1 + u % 7, c = u / 7;
    tk = q < 8 ? rank * 8 + q : ((rank + i) & 7) * 8 + ((c + SKEW * i) & 7);
  }
  const unsigned chunk = self ? 0 : (kt - SKEW * ((rank - (int)dest) & 7)) & 7;
  const unsigned send = chunk == 0 ? 0x40000000u : chunk == 1 ? grp - 6 : 2 * 1 * (chunk - 2) + grp - 2;
  const unsigned abase = WS_A + (mh * 16 + w * 4) * 1024;
  const unsigned dbase = WS_A + (rank * 8 + kt) * 65536 + m16s * 1024;
  const unsigned fdst = WS_FLAG + ((rank * 8 + kt) * 8 + xcd) * 128;
  const unsigned pub = w == 0 ? 2 : 0;
  const unsigned xloc = (unsigned)(rank * 512 + mh * 256 + w * 64) * 2048u;
  const unsigned fsrc = WS_FLAG + (mh * 4 + w) * 128;
  const unsigned cnt = WS_CNT + b * 128, tmo = WS_TMO;
  const unsigned lr = ldsb + 32768, lw = lr + w * 4096, lx = ldsb + w * 32768;
  const unsigned dbg2 = wave == 4 ? (12u << 20) + b * 256 : 0x7ff00000u;
  unsigned vc = (lane & 31) * 128 + (lane >> 5) * 32768;
  const unsigned cbase = (unsigned)(((mh * 256 + w * 64) * N_ + n0) * 4), pj = (j >> 1) & 15;
  asm volatile(
      "s_nop 4\n\t"
      "buffer_load_dword v16, off, %[srds], %[cnt] sc1\n\t"
      "s_memrealtime s[88:89]\n\t"
      "s_waitcnt lgkmcnt(0)\n\t"
      "s_mov_b32 s75, s88\n\t"
      "s_mov_b32 s82, 0\n\t"
      "s_add_u32 s76, s88, 0\n\t"
      "s_mov_b32 s71, s76\n\t"
      "s_mov_b32 s85, 222\n\t"
      "s_mov_b32 s83, 0\n\t"
      "s_mov_b32 s97, 1\n\t"
      "s_mov_b32 s81, %[send]\n\t"
      "s_xor_b32 s78, %[grp], 1\n\t"
      "v_readlane_b32 s72, %[tk], s78\n\t"
      "s_nop 3\n\t"
      "s_lshl_b32 s72, s72, 22\n\t"
      "s_cmp_lt_u32 s78, 64\n\t"
      "s_cselect_b32 s72, s72, 0x80000000\n\t"
      "s_cmp_eq_u32 %[chunk], 0\n\t"
      "s_cbranch_scc1 23f\n\t"
      "s_sleep 20\n\t"
      "23:\n\t"
      "s_cmp_gt_u32 %[chunk], 1\n\t"
      "s_cbranch_scc1 21f\n\t"
      "buffer_load_dwordx4 v[0:3], %[vx], %[srdx], 0 offen offset:0\n\t"
      "buffer_load_dwordx4 v[4:7], %[vx], %[srdx], 0 offen offset:64\n\t"
      "buffer_load_dwordx4 v[8:11], %[vx], %[srdx], 0 offen offset:128\n\t"
      "buffer_load_dwordx4 v[12:15], %[vx], %[srdx], 0 offen offset:192\n\t"
      "s_cmp_lg_u32 %[chunk], 0\n\t"
      "s_cbranch_scc1 24f\n\t"
      "s_waitcnt vmcnt(4)\n\t"
      "v_readfirstlane_b32 s79, v16\n\t"
      "s_nop 3\n\t"
      "s_add_u32 s79, s79, 1\n\t"
      "s_and_b32 s77, s79, 1\n\t"
      "s_lshl_b32 s78, s77, 22\n\t"
      "s_add_u32 s80, s78, %[abase]\n\t"
      "s_add_u32 s84, s78, %[dbase]\n\t"
      "s_waitcnt vmcnt(0)\n\t"
      "v_cvt_pk_f16_f32 v240, v0, v1\n\t"
      "v_cvt_pk_f16_f32 v241, v2, v3\n\t"
      "v_cvt_pk_f16_f32 v242, v4, v5\n\t"
      "v_cvt_pk_f16_f32 v243, v6, v7\n\t"
      "v_cvt_pk_f16_f32 v244, v8, v9\n\t"
      "v_cvt_pk_f16_f32 v245, v10, v11\n\t"
      "v_cvt_pk_f16_f32 v246, v12, v13\n\t"
      "v_cvt_pk_f16_f32 v247, v14, v15\n\t"
      "buffer_store_dwordx4 v[240:243], %[v16], %[srdd], s84 offen sc0 sc1\n\t"
      "s_add_u32 s77, s84, 0x8000\n\t"
      "buffer_store_dwordx4 v[244:247], %[v16], %[srdd], s77 offen sc0 sc1\n\t"
      "s_waitcnt vmcnt(0)\n\t"
      "s_barrier\n\t"
      "s_cmp_eq_u32 %[pub], 0\n\t"
      "s_cbranch_scc1 26f\n\t"
      "v_mov_b32 v240, s79\n\t"
      "s_mov_b64 exec, 1\n\t"
      "buffer_store_dword v240, off, %[srdd], %[fdst] sc0 sc1\n\t"
      "s_mov_b64 exec, -1\n\t"
      "26:\n\t"
      "buffer_load_dwordx4 v[192:195], %[vw], %[srdw], s72 offen\n\t"
      "s_add_u32 s72, s72, 65536\n\t"
      "buffer_load_dwordx4 v[196:199], %[vw], %[srdw], s72 offen\n\t"
      "s_add_u32 s72, s72, 65536\n\t"
      "buffer_load_dwordx4 v[200:203], %[vw], %[srdw], s72 offen\n\t"
      "s_add_u32 s72, s72, 65536\n\t"
      "buffer_load_dwordx4 v[204:207], %[vw], %[srdw], s72 offen\n\t"
      "s_add_u32 s72, s72, 851968\n\t"
      "buffer_load_dwordx4 v[208:211], %[vw], %[srdw], s72 offen\n\t"
      "s_add_u32 s72, s72, 65536\n\t"
      "buffer_load_dwordx4 v[212:215], %[vw], %[srdw], s72 offen\n\t"
      "s_add_u32 s72, s72, 65536\n\t"
      "buffer_load_dwordx4 v[216:219], %[vw], %[srdw], s72 offen\n\t"
      "s_add_u32 s72, s72, 65536\n\t"
      "buffer_load_dwordx4 v[220:223], %[vw], %[srdw], s72 offen\n\t"
      "s_branch 22f\n\t"
      "24:\n\t"
      "buffer_load_dwordx4 v[192:195], %[vw], %[srdw], s72 offen\n\t"
      "s_add_u32 s72, s72, 65536\n\t"
      "buffer_load_dwordx4 v[196:199], %[vw], %[srdw], s72 offen\n\t"
      "s_add_u32 s72, s72, 65536\n\t"
      "buffer_load_dwordx4 v[200:203], %[vw], %[srdw], s72 offen\n\t"
      "s_add_u32 s72, s72, 65536\n\t"
      "buffer_load_dwordx4 v[204:207], %[vw], %[srdw], s72 offen\n\t"
      "s_add_u32 s72, s72, 851968\n\t"
      "buffer_load_dwordx4 v[208:211], %[vw], %[srdw], s72 offen\n\t"
      "s_add_u32 s72, s72, 65536\n\t"
      "buffer_load_dwordx4 v[212:215], %[vw], %[srdw], s72 offen\n\t"
      "s_add_u32 s72, s72, 65536\n\t"
      "buffer_load_dwordx4 v[216:219], %[vw], %[srdw], s72 offen\n\t"
      "s_add_u32 s72, s72, 65536\n\t"
      "buffer_load_dwordx4 v[220:223], %[vw], %[srdw], s72 offen\n\t"
      "s_barrier\n\t"
      "s_waitcnt vmcnt(8)\n\t"
      "v_readfirstlane_b32 s79, v16\n\t"
      "s_nop 3\n\t"
      "s_add_u32 s79, s79, 1\n\t"
      "s_and_b32 s77, s79, 1\n\t"
      "s_lshl_b32 s78, s77, 22\n\t"
      "s_add_u32 s80, s78, %[abase]\n\t"
      "s_add_u32 s84, s78, %[dbase]\n\t"
      "v_cvt_pk_f16_f32 v240, v0, v1\n\t"
      "v_cvt_pk_f16_f32 v241, v2, v3\n\t"
      "v_cvt_pk_f16_f32 v242, v4, v5\n\t"
      "v_cvt_pk_f16_f32 v243, v6, v7\n\t"
      "v_cvt_pk_f16_f32 v244, v8, v9\n\t"
      "v_cvt_pk_f16_f32 v245, v10, v11\n\t"
      "v_cvt_pk_f16_f32 v246, v12, v13\n\t"
      "v_cvt_pk_f16_f32 v247, v14, v15\n\t"
      "s_branch 22f\n\t"
      "21:\n\t"
      "buffer_load_dwordx4 v[192:195], %[vw], %[srdw], s72 offen\n\t"
      "s_add_u32 s72, s72, 65536\n\t"
      "buffer_load_dwordx4 v[196:199], %[vw], %[srdw], s72 offen\n\t"
      "s_add_u32 s72, s72, 65536\n\t"
      "buffer_load_dwordx4 v[200:203], %[vw], %[srdw], s72 offen\n\t"
      "s_add_u32 s72, s72, 65536\n\t"
      "buffer_load_dwordx4 v[204:207], %[vw], %[srdw], s72 offen\n\t"
      "s_add_u32 s72, s72, 851968\n\t"
      "buffer_load_dwordx4 v[208:211], %[vw], %[srdw], s72 offen\n\t"
      "s_add_u32 s72, s72, 65536\n\t"
      "buffer_load_dwordx4 v[212:215], %[vw], %[srdw], s72 offen\n\t"
      "s_add_u32 s72, s72, 65536\n\t"
      "buffer_load_dwordx4 v[216:219], %[vw], %[srdw], s72 offen\n\t"
      "s_add_u32 s72, s72, 65536\n\t"
      "buffer_load_dwordx4 v[220:223], %[vw], %[srdw], s72 offen\n\t"
      "s_barrier\n\t"
      "s_waitcnt vmcnt(8)\n\t"
      "v_readfirstlane_b32 s79, v16\n\t"
      "s_nop 3\n\t"
      "s_add_u32 s79, s79, 1\n\t"
      "s_and_b32 s77, s79, 1\n\t"
      "s_lshl_b32 s78, s77, 22\n\t"
      "s_add_u32 s80, s78, %[abase]\n\t"
      "s_add_u32 s84, s78, %[dbase]\n\t"
      "22:\n\t"
      "v_mov_b32 v0, 0\n\t"
      "v_mov_b32 v1, 0\n\t"
      "v_mov_b32 v2, 0\n\t"
      "v_mov_b32 v3, 0\n\t"
      "v_mov_b32 v4, 0\n\t"
      "v_mov_b32 v5, 0\n\t"
      "v_mov_b32 v6, 0\n\t"
      "v_mov_b32 v7, 0\n\t"
      "v_mov_b32 v8, 0\n\t"
      "v_mov_b32 v9, 0\n\t"
      "v_mov_b32 v10, 0\n\t"
      "v_mov_b32 v11, 0\n\t"
      "v_mov_b32 v12, 0\n\t"
      "v_mov_b32 v13, 0\n\t"
      "v_mov_b32 v14, 0\n\t"
      "v_mov_b32 v15, 0\n\t"
      "v_mov_b32 v16, 0\n\t"
      "v_mov_b32 v17, 0\n\t"
      "v_mov_b32 v18, 0\n\t"
      "v_mov_b32 v19, 0\n\t"
      "v_mov_b32 v20, 0\n\t"
      "v_mov_b32 v21, 0\n\t"
      "v_mov_b32 v22, 0\n\t"
      "v_mov_b32 v23, 0\n\t"
      "v_mov_b32 v24, 0\n\t"
      "v_mov_b32 v25, 0\n\t"
      "v_mov_b32 v26, 0\n\t"
      "v_mov_b32 v27, 0\n\t"
      "v_mov_b32 v28, 0\n\t"
      "v_mov_b32 v29, 0\n\t"
      "v_mov_b32 v30, 0\n\t"
      "v_mov_b32 v31, 0\n\t"
      "v_mov_b32 v32, 0\n\t"
      "v_mov_b32 v33, 0\n\t"
      "v_mov_b32 v34, 0\n\t"
      "v_mov_b32 v35, 0\n\t"
      "v_mov_b32 v36, 0\n\t"
      "v_mov_b32 v37, 0\n\t"
      "v_mov_b32 v38, 0\n\t"
      "v_mov_b32 v39, 0\n\t"
      "v_mov_b32 v40, 0\n\t"
      "v_mov_b32 v41, 0\n\t"
      "v_mov_b32 v42, 0\n\t"
      "v_mov_b32 v43, 0\n\t"
      "v_mov_b32 v44, 0\n\t"
      "v_mov_b32 v45, 0\n\t"
      "v_mov_b32 v46, 0\n\t"
      "v_mov_b32 v47, 0\n\t"
      "v_mov_b32 v48, 0\n\t"
      "v_mov_b32 v49, 0\n\t"
      "v_mov_b32 v50, 0\n\t"
      "v_mov_b32 v51, 0\n\t"
      "v_mov_b32 v52, 0\n\t"
      "v_mov_b32 v53, 0\n\t"
      "v_mov_b32 v54, 0\n\t"
      "v_mov_b32 v55, 0\n\t"
      "v_mov_b32 v56, 0\n\t"
      "v_mov_b32 v57, 0\n\t"
      "v_mov_b32 v58, 0\n\t"
      "v_mov_b32 v59, 0\n\t"
      "v_mov_b32 v60, 0\n\t"
      "v_mov_b32 v61, 0\n\t"
      "v_mov_b32 v62, 0\n\t"
      "v_mov_b32 v63, 0\n\t"
      "v_mov_b32 v64, 0\n\t"
      "v_mov_b32 v65, 0\n\t"
      "v_mov_b32 v66, 0\n\t"
      "v_mov_b32 v67, 0\n\t"
      "v_mov_b32 v68, 0\n\t"
      "v_mov_b32 v69, 0\n\t"
      "v_mov_b32 v70, 0\n\t"
      "v_mov_b32 v71, 0\n\t"
      "v_mov_b32 v72, 0\n\t"
      "v_mov_b32 v73, 0\n\t"
      "v_mov_b32 v74, 0\n\t"
      "v_mov_b32 v75, 0\n\t"
      "v_mov_b32 v76, 0\n\t"
      "v_mov_b32 v77, 0\n\t"
      "v_mov_b32 v78, 0\n\t"
      "v_mov_b32 v79, 0\n\t"
      "v_mov_b32 v80, 0\n\t"
      "v_mov_b32 v81, 0\n\t"
      "v_mov_b32 v82, 0\n\t"
      "v_mov_b32 v83, 0\n\t"
      "v_mov_b32 v84, 0\n\t"
      "v_mov_b32 v85, 0\n\t"
      "v_mov_b32 v86, 0\n\t"
      "v_mov_b32 v87, 0\n\t"
      "v_mov_b32 v88, 0\n\t"
      "v_mov_b32 v89, 0\n\t"
      "v_mov_b32 v90, 0\n\t"
      "v_mov_b32 v91, 0\n\t"
      "v_mov_b32 v92, 0\n\t"
      "v_mov_b32 v93, 0\n\t"
      "v_mov_b32 v94, 0\n\t"
      "v_mov_b32 v95, 0\n\t"
      "v_mov_b32 v96, 0\n\t"
      "v_mov_b32 v97, 0\n\t"
      "v_mov_b32 v98, 0\n\t"
      "v_mov_b32 v99, 0\n\t"
      "v_mov_b32 v100, 0\n\t"
      "v_mov_b32 v101, 0\n\t"
      "v_mov_b32 v102, 0\n\t"
      "v_mov_b32 v103, 0\n\t"
      "v_mov_b32 v104, 0\n\t"
      "v_mov_b32 v105, 0\n\t"
      "v_mov_b32 v106, 0\n\t"
      "v_mov_b32 v107, 0\n\t"
      "v_mov_b32 v108, 0\n\t"
      "v_mov_b32 v109, 0\n\t"
      "v_mov_b32 v110, 0\n\t"
      "v_mov_b32 v111, 0\n\t"
      "v_mov_b32 v112, 0\n\t"
      "v_mov_b32 v113, 0\n\t"
      "v_mov_b32 v114, 0\n\t"
      "v_mov_b32 v115, 0\n\t"
      "v_mov_b32 v116, 0\n\t"
      "v_mov_b32 v117, 0\n\t"
      "v_mov_b32 v118, 0\n\t"
      "v_mov_b32 v119, 0\n\t"
      "v_mov_b32 v120, 0\n\t"
      "v_mov_b32 v121, 0\n\t"
      "v_mov_b32 v122, 0\n\t"
      "v_mov_b32 v123, 0\n\t"
      "v_mov_b32 v124, 0\n\t"
      "v_mov_b32 v125, 0\n\t"
      "v_mov_b32 v126, 0\n\t"
      "v_mov_b32 v127, 0\n\t"
      "s_xor_b32 s78, %[grp], 1\n\t"
      "s_and_b32 s77, s78, 7\n\t"
      "s_lshl_b32 s77, s77, 14\n\t"
      "s_add_u32 s77, s77, %[lw]\n\t"
      "v_add_u32 v255, s77, %[v16]\n\t"
      "s_xor_b32 s78, %[grp], 1\n\t"
      "s_add_u32 s78, s78, 2\n\t"
      "v_readlane_b32 s72, %[tk], s78\n\t"
      "s_nop 3\n\t"
      "s_lshl_b32 s72, s72, 22\n\t"
      "s_cmp_lt_u32 s78, 64\n\t"
      "s_cselect_b32 s72, s72, 0x80000000\n\t"
      "s_waitcnt vmcnt(0)\n\t"
      "v_cvt_pk_f16_f32 v224, v192, v196\n\t"
      "v_cvt_pk_f16_f32 v225, v200, v204\n\t"
      "v_cvt_pk_f16_f32 v226, v208, v212\n\t"
      "v_cvt_pk_f16_f32 v227, v216, v220\n\t"
      "ds_write_b128 v255, v[224:227] offset:0\n\t"
      "v_cvt_pk_f16_f32 v228, v193, v197\n\t"
      "v_cvt_pk_f16_f32 v229, v201, v205\n\t"
      "v_cvt_pk_f16_f32 v230, v209, v213\n\t"
      "v_cvt_pk_f16_f32 v231, v217, v221\n\t"
      "ds_write_b128 v255, v[228:231] offset:1024\n\t"
      "v_cvt_pk_f16_f32 v232, v194, v198\n\t"
      "v_cvt_pk_f16_f32 v233, v202, v206\n\t"
      "v_cvt_pk_f16_f32 v234, v210, v214\n\t"
      "v_cvt_pk_f16_f32 v235, v218, v222\n\t"
      "ds_write_b128 v255, v[232:235] offset:2048\n\t"
      "v_cvt_pk_f16_f32 v236, v195, v199\n\t"
      "v_cvt_pk_f16_f32 v237, v203, v207\n\t"
      "v_cvt_pk_f16_f32 v238, v211, v215\n\t"
      "v_cvt_pk_f16_f32 v239, v219, v223\n\t"
      "ds_write_b128 v255, v[236:239] offset:3072\n\t"
      "buffer_load_dwordx4 v[192:195], %[vw], %[srdw], s72 offen\n\t"
      "s_add_u32 s72, s72, 65536\n\t"
      "buffer_load_dwordx4 v[196:199], %[vw], %[srdw], s72 offen\n\t"
      "s_add_u32 s72, s72, 65536\n\t"
      "buffer_load_dwordx4 v[200:203], %[vw], %[srdw], s72 offen\n\t"
      "s_add_u32 s72, s72, 65536\n\t"
      "buffer_load_dwordx4 v[204:207], %[vw], %[srdw], s72 offen\n\t"
      "s_add_u32 s72, s72, 851968\n\t"
      "buffer_load_dwordx4 v[208:211], %[vw], %[srdw], s72 offen\n\t"
      "s_add_u32 s72, s72, 65536\n\t"
      "buffer_load_dwordx4 v[212:215], %[vw], %[srdw], s72 offen\n\t"
      "s_add_u32 s72, s72, 65536\n\t"
      "buffer_load_dwordx4 v[216:219], %[vw], %[srdw], s72 offen\n\t"
      "s_add_u32 s72, s72, 65536\n\t"
      "buffer_load_dwordx4 v[220:223], %[vw], %[srdw], s72 offen\n\t"
      "s_xor_b32 s78, %[grp], 1\n\t"
      "s_add_u32 s78, s78, 2\n\t"
      "s_and_b32 s77, s78, 7\n\t"
      "s_lshl_b32 s77, s77, 14\n\t"
      "s_add_u32 s77, s77, %[lw]\n\t"
      "v_add_u32 v255, s77, %[v16]\n\t"
      "s_xor_b32 s78, %[grp], 1\n\t"
      "s_add_u32 s78, s78, 4\n\t"
      "v_readlane_b32 s72, %[tk], s78\n\t"
      "s_nop 3\n\t"
      "s_lshl_b32 s72, s72, 22\n\t"
      "s_cmp_lt_u32 s78, 64\n\t"
      "s_cselect_b32 s72, s72, 0x80000000\n\t"
      "s_waitcnt vmcnt(0) lgkmcnt(0)\n\t"
      "v_cvt_pk_f16_f32 v224, v192, v196\n\t"
      "v_cvt_pk_f16_f32 v225, v200, v204\n\t"
      "v_cvt_pk_f16_f32 v226, v208, v212\n\t"
      "v_cvt_pk_f16_f32 v227, v216, v220\n\t"
      "ds_write_b128 v255, v[224:227] offset:0\n\t"
      "v_cvt_pk_f16_f32 v228, v193, v197\n\t"
      "v_cvt_pk_f16_f32 v229, v201, v205\n\t"
      "v_cvt_pk_f16_f32 v230, v209, v213\n\t"
      "v_cvt_pk_f16_f32 v231, v217, v221\n\t"
      "ds_write_b128 v255, v[228:231] offset:1024\n\t"
      "v_cvt_pk_f16_f32 v232, v194, v198\n\t"
      "v_cvt_pk_f16_f32 v233, v202, v206\n\t"
      "v_cvt_pk_f16_f32 v234, v210, v214\n\t"
      "v_cvt_pk_f16_f32 v235, v218, v222\n\t"
      "ds_write_b128 v255, v[232:235] offset:2048\n\t"
      "v_cvt_pk_f16_f32 v236, v195, v199\n\t"
      "v_cvt_pk_f16_f32 v237, v203, v207\n\t"
      "v_cvt_pk_f16_f32 v238, v211, v215\n\t"
      "v_cvt_pk_f16_f32 v239, v219, v223\n\t"
      "ds_write_b128 v255, v[236:239] offset:3072\n\t"
      "s_waitcnt lgkmcnt(0)\n\t"
      "s_barrier\n\t"
      "s_cmp_lg_u32 %[lw], %[lr]\n\t"
      "s_cbranch_scc1 27f\n\t"
      "s_cmp_lg_u32 %[grp], 0\n\t"
      "s_cbranch_scc1 27f\n\t"
      "v_mov_b32 v255, s79\n\t"
      "s_mov_b64 exec, 1\n\t"
      "buffer_store_dword v255, off, %[srds], %[cnt] sc1\n\t"
      "s_mov_b64 exec, -1\n\t"
      "s_nop 1\n\t"
      "27:\n\t"
      "s_mov_b64 s[90:91], 1\n\t"
      "s_lshl_b64 s[90:91], s[90:91], %[grp]\n\t"
      "s_cmp_lg_u32 s83, 0\n\t"
      "s_cbranch_scc1 29f\n\t"
      "s_memrealtime s[86:87]\n\t"
      "28:\n\t"
      "v_lshlrev_b32 v255, 10, %[tk]\n\t"
      "s_mov_b64 exec, s[90:91]\n\t"
      "buffer_load_dword %[vp], v255, %[srds], %[fsrc] offen sc0 sc1\n\t"
      "s_mov_b64 exec, -1\n\t"
      "s_memrealtime s[88:89]\n\t"
      "s_waitcnt vmcnt(0) lgkmcnt(0)\n\t"
      "v_cmp_le_u32 s[92:93], s79, %[vp]\n\t"
      "s_sub_u32 s88, s88, s86\n\t"
      "s_nop 3\n\t"
      "s_andn2_b64 s[90:91], s[90:91], s[92:93]\n\t"
      "s_cbranch_scc0 29f\n\t"
      "s_cmp_lt_u32 s88, 10000\n\t"
      "s_cbranch_scc1 31f\n\t"
      "buffer_load_dword v255, off, %[srds], %[tmo] sc1\n\t"
      "s_waitcnt vmcnt(0)\n\t"
      "v_readfirstlane_b32 s77, v255\n\t"
      "s_nop 3\n\t"
      "s_cmp_eq_u32 s77, s79\n\t"
      "s_cbranch_scc1 30f\n\t"
      "s_cmp_gt_u32 s88, 0x2000000\n\t"
      "s_cbranch_scc1 30f\n\t"
      "s_sleep 64\n\t"
      "31:\n\t"
      "s_sleep 4\n\t"
      "s_branch 28b\n\t"
      "30:\n\t"
      "s_mov_b32 s83, 1\n\t"
      "v_mov_b32 v255, s79\n\t"
      "s_mov_b64 exec, 1\n\t"
      "buffer_store_dword v255, off, %[srds], %[tmo] sc1\n\t"
      "s_mov_b64 exec, -1\n\t"
      "29:\n\t"
      "s_mov_b32 s78, %[grp]\n\t"
      "v_readlane_b32 s73, %[tk], s78\n\t"
      "s_nop 3\n\t"
      "s_lshl_b32 s73, s73, 16\n\t"
      "s_add_u32 s73, s73, s80\n\t"
      "s_cmp_lt_u32 s78, 64\n\t"
      "s_cselect_b32 s73, s73, 0x80000000\n\t"
      "s_add_u32 s74, s73, 0x8000\n\t"
      "buffer_load_dwordx4 v[128:131], %[v16], %[srds], s73 offen sc0 sc1\n\t"
      "buffer_load_dwordx4 v[132:135], %[v16], %[srds], s73 offen offset:1024 sc0 sc1\n\t"
      "buffer_load_dwordx4 v[136:139], %[v16], %[srds], s73 offen offset:2048 sc0 sc1\n\t"
      "buffer_load_dwordx4 v[140:143], %[v16], %[srds], s73 offen offset:3072 sc0 sc1\n\t"
      "buffer_load_dwordx4 v[144:147], %[v16], %[srds], s74 offen sc0 sc1\n\t"
      "buffer_load_dwordx4 v[148:151], %[v16], %[srds], s74 offen offset:1024 sc0 sc1\n\t"
      "buffer_load_dwordx4 v[152:155], %[v16], %[srds], s74 offen offset:2048 sc0 sc1\n\t"
      "buffer_load_dwordx4 v[156:159], %[v16], %[srds], s74 offen offset:3072 sc0 sc1\n\t"
      "s_mov_b64 s[90:91], 0x54\n\t"
      "s_lshl_b64 s[90:91], s[90:91], %[grp]\n\t"
      "s_cmp_lg_u32 s83, 0\n\t"
      "s_cbranch_scc1 33f\n\t"
      "s_memrealtime s[86:87]\n\t"
      "32:\n\t"
      "v_lshlrev_b32 v255, 10, %[tk]\n\t"
      "s_mov_b64 exec, s[90:91]\n\t"
      "buffer_load_dword %[vp], v255, %[srds], %[fsrc] offen sc0 sc1\n\t"
      "s_mov_b64 exec, -1\n\t"
      "s_memrealtime s[88:89]\n\t"
      "s_waitcnt vmcnt(0) lgkmcnt(0)\n\t"
      "v_cmp_le_u32 s[92:93], s79, %[vp]\n\t"
      "s_sub_u32 s88, s88, s86\n\t"
      "s_nop 3\n\t"
      "s_andn2_b64 s[90:91], s[90:91], s[92:93]\n\t"
      "s_cbranch_scc0 33f\n\t"
      "s_cmp_lt_u32 s88, 10000\n\t"
      "s_cbranch_scc1 35f\n\t"
      "buffer_load_dword v255, off, %[srds], %[tmo] sc1\n\t"
      "s_waitcnt vmcnt(0)\n\t"
      "v_readfirstlane_b32 s77, v255\n\t"
      "s_nop 3\n\t"
      "s_cmp_eq_u32 s77, s79\n\t"
      "s_cbranch_scc1 34f\n\t"
      "s_cmp_gt_u32 s88, 0x2000000\n\t"
      "s_cbranch_scc1 34f\n\t"
      "s_sleep 64\n\t"
      "35:\n\t"
      "s_sleep 4\n\t"
      "s_branch 32b\n\t"
      "34:\n\t"
      "s_mov_b32 s83, 1\n\t"
      "v_mov_b32 v255, s79\n\t"
      "s_mov_b64 exec, 1\n\t"
      "buffer_store_dword v255, off, %[srds], %[tmo] sc1\n\t"
      "s_mov_b64 exec, -1\n\t"
      "33:\n\t"
      "buffer_load_dwordx4 v[192:195], %[vw], %[srdw], s72 offen\n\t"
      "s_add_u32 s72, s72, 65536\n\t"
      "buffer_load_dwordx4 v[196:199], %[vw], %[srdw], s72 offen\n\t"
      "s_add_u32 s72, s72, 65536\n\t"
      "buffer_load_dwordx4 v[200:203], %[vw], %[srdw], s72 offen\n\t"
      "s_add_u32 s72, s72, 65536\n\t"
      "buffer_load_dwordx4 v[204:207], %[vw], %[srdw], s72 offen\n\t"
      "s_add_u32 s72, s72, 851968\n\t"
      "buffer_load_dwordx4 v[208:211], %[vw], %[srdw], s72 offen\n\t"
      "s_add_u32 s72, s72, 65536\n\t"
      "buffer_load_dwordx4 v[212:215], %[vw], %[srdw], s72 offen\n\t"
      "s_add_u32 s72, s72, 65536\n\t"
      "buffer_load_dwordx4 v[216:219], %[vw], %[srdw], s72 offen\n\t"
      "s_add_u32 s72, s72, 65536\n\t"
      "buffer_load_dwordx4 v[220:223], %[vw], %[srdw], s72 offen\n\t"
      "s_add_u32 s78, %[grp], 2\n\t"
      "v_readlane_b32 s73, %[tk], s78\n\t"
      "s_nop 3\n\t"
      "s_lshl_b32 s73, s73, 16\n\t"
      "s_add_u32 s73, s73, s80\n\t"
      "s_cmp_lt_u32 s78, 64\n\t"
      "s_cselect_b32 s73, s73, 0x80000000\n\t"
      "s_add_u32 s74, s73, 0x8000\n\t"
      "buffer_load_dwordx4 v[160:163], %[v16], %[srds], s73 offen sc0 sc1\n\t"
      "buffer_load_dwordx4 v[164:167], %[v16], %[srds], s73 offen offset:1024 sc0 sc1\n\t"
      "buffer_load_dwordx4 v[168:171], %[v16], %[srds], s73 offen offset:2048 sc0 sc1\n\t"
      "buffer_load_dwordx4 v[172:175], %[v16], %[srds], s73 offen offset:3072 sc0 sc1\n\t"
      "buffer_load_dwordx4 v[176:179], %[v16], %[srds], s74 offen sc0 sc1\n\t"
      "buffer_load_dwordx4 v[180:183], %[v16], %[srds], s74 offen offset:1024 sc0 sc1\n\t"
      "buffer_load_dwordx4 v[184:187], %[v16], %[srds], s74 offen offset:2048 sc0 sc1\n\t"
      "buffer_load_dwordx4 v[188:191], %[v16], %[srds], s74 offen offset:3072 sc0 sc1\n\t"
      "s_cmp_lg_u32 %[chunk], 1\n\t"
      "s_cbranch_scc1 36f\n\t"
      "buffer_store_dwordx4 v[240:243], %[v16], %[srdd], s84 offen sc0 sc1\n\t"
      "s_add_u32 s77, s84, 0x8000\n\t"
      "buffer_store_dwordx4 v[244:247], %[v16], %[srdd], s77 offen sc0 sc1\n\t"
      "s_branch 37f\n\t"
      "36:\n\t"
      "s_sub_u32 s70, %[grp], 2\n\t"
      "s_sub_u32 s77, s70, s81\n\t"
      "s_cmp_gt_u32 s77, 8\n\t"
      "s_cbranch_scc1 38f\n\t"
      "s_cmp_eq_u32 s77, 2\n\t"
      "s_cbranch_scc1 39f\n\t"
      "s_cmp_eq_u32 s77, 4\n\t"
      "s_cbranch_scc1 40f\n\t"
      "s_cmp_eq_u32 s77, 8\n\t"
      "s_cbranch_scc1 41f\n\t"
      "s_cmp_lg_u32 s77, 0\n\t"
      "s_cbranch_scc1 38f\n\t"
      "buffer_load_dwordx4 v[240:243], %[vx], %[srdx], 0 offen offset:0 nt\n\t"
      "buffer_load_dwordx4 v[244:247], %[vx], %[srdx], 0 offen offset:64 nt\n\t"
      "s_branch 38f\n\t"
      "39:\n\t"
      "v_cvt_pk_f16_f32 v240, v240, v241\n\t"
      "v_cvt_pk_f16_f32 v241, v242, v243\n\t"
      "v_cvt_pk_f16_f32 v242, v244, v245\n\t"
      "v_cvt_pk_f16_f32 v243, v246, v247\n\t"
      "buffer_store_dwordx4 v[240:243], %[v16], %[srdd], s84 offen sc0 sc1\n\t"
      "s_nop 1\n\t"
      "buffer_load_dwordx4 v[240:243], %[vx], %[srdx], 0 offen offset:128 nt\n\t"
      "buffer_load_dwordx4 v[244:247], %[vx], %[srdx], 0 offen offset:192 nt\n\t"
      "s_branch 38f\n\t"
      "40:\n\t"
      "v_cvt_pk_f16_f32 v240, v240, v241\n\t"
      "v_cvt_pk_f16_f32 v241, v242, v243\n\t"
      "v_cvt_pk_f16_f32 v242, v244, v245\n\t"
      "v_cvt_pk_f16_f32 v243, v246, v247\n\t"
      "s_add_u32 s77, s84, 0x8000\n\t"
      "buffer_store_dwordx4 v[240:243], %[v16], %[srdd], s77 offen sc0 sc1\n\t"
      "s_branch 38f\n\t"
      "41:\n\t"
      "s_cmp_eq_u32 %[pub], 0\n\t"
      "s_cbranch_scc1 38f\n\t"
      "v_mov_b32 v240, s79\n\t"
      "s_mov_b64 exec, 1\n\t"
      "buffer_store_dword v240, off, %[srdd], %[fdst] sc0 sc1\n\t"
      "s_mov_b64 exec, -1\n\t"
      "38:\n\t"
      "37:\n\t"
      "s_cmp_eq_u32 %[grp], 0\n\t"
      "s_cbranch_scc1 3f\n\t"
      "s_mov_b32 s78, 6\n\t"
      "v_readlane_b32 s72, %[tk], s78\n\t"
      "s_nop 3\n\t"
      "s_lshl_b32 s72, s72, 22\n\t"
      "s_cmp_lt_u32 s78, 64\n\t"
      "s_cselect_b32 s72, s72, 0x80000000\n\t"
      "s_mov_b32 s78, 4\n\t"
      "s_and_b32 s77, s78, 7\n\t"
      "s_lshl_b32 s77, s77, 14\n\t"
      "s_add_u32 s77, s77, %[lw]\n\t"
      "v_add_u32 v255, s77, %[v16]\n\t"
      "s_waitcnt vmcnt(8)\n\t"
      "v_cvt_pk_f16_f32 v224, v192, v196\n\t"
      "v_cvt_pk_f16_f32 v225, v200, v204\n\t"
      "v_cvt_pk_f16_f32 v226, v208, v212\n\t"
      "v_cvt_pk_f16_f32 v227, v216, v220\n\t"
      "ds_write_b128 v255, v[224:227] offset:0\n\t"
      "v_cvt_pk_f16_f32 v228, v193, v197\n\t"
      "v_cvt_pk_f16_f32 v229, v201, v205\n\t"
      "v_cvt_pk_f16_f32 v230, v209, v213\n\t"
      "v_cvt_pk_f16_f32 v231, v217, v221\n\t"
      "ds_write_b128 v255, v[228:231] offset:1024\n\t"
      "v_cvt_pk_f16_f32 v232, v194, v198\n\t"
      "v_cvt_pk_f16_f32 v233, v202, v206\n\t"
      "v_cvt_pk_f16_f32 v234, v210, v214\n\t"
      "v_cvt_pk_f16_f32 v235, v218, v222\n\t"
      "ds_write_b128 v255, v[232:235] offset:2048\n\t"
      "v_cvt_pk_f16_f32 v236, v195, v199\n\t"
      "v_cvt_pk_f16_f32 v237, v203, v207\n\t"
      "v_cvt_pk_f16_f32 v238, v211, v215\n\t"
      "v_cvt_pk_f16_f32 v239, v219, v223\n\t"
      "ds_write_b128 v255, v[236:239] offset:3072\n\t"
      "buffer_load_dwordx4 v[192:195], %[vw], %[srdw], s72 offen\n\t"
      "s_add_u32 s72, s72, 65536\n\t"
      "buffer_load_dwordx4 v[196:199], %[vw], %[srdw], s72 offen\n\t"
      "s_add_u32 s72, s72, 65536\n\t"
      "buffer_load_dwordx4 v[200:203], %[vw], %[srdw], s72 offen\n\t"
      "s_add_u32 s72, s72, 65536\n\t"
      "buffer_load_dwordx4 v[204:207], %[vw], %[srdw], s72 offen\n\t"
      "s_add_u32 s72, s72, 851968\n\t"
      "buffer_load_dwordx4 v[208:211], %[vw], %[srdw], s72 offen\n\t"
      "s_add_u32 s72, s72, 65536\n\t"
      "buffer_load_dwordx4 v[212:215], %[vw], %[srdw], s72 offen\n\t"
      "s_add_u32 s72, s72, 65536\n\t"
      "buffer_load_dwordx4 v[216:219], %[vw], %[srdw], s72 offen\n\t"
      "s_add_u32 s72, s72, 65536\n\t"
      "buffer_load_dwordx4 v[220:223], %[vw], %[srdw], s72 offen\n\t"
      "s_mov_b32 s70, 1\n\t"
      "s_and_b32 s77, s70, 7\n\t"
      "s_lshl_b32 s77, s77, 14\n\t"
      "s_add_u32 s77, s77, %[lr]\n\t"
      "v_add_u32 v254, s77, %[v16]\n\t"
      "s_waitcnt lgkmcnt(0)\n\t"
      "ds_read_b128 v[224:227], v254 offset:0\n\t"
      "ds_read_b128 v[228:231], v254 offset:8192\n\t"
      "s_barrier\n\t"
      "s_branch 4f\n\t"
      "3:\n\t"
      "s_mov_b32 s70, 0\n\t"
      "s_and_b32 s77, s70, 7\n\t"
      "s_lshl_b32 s77, s77, 14\n\t"
      "s_add_u32 s77, s77, %[lr]\n\t"
      "v_add_u32 v254, s77, %[v16]\n\t"
      "ds_read_b128 v[224:227], v254 offset:0\n\t"
      "ds_read_b128 v[228:231], v254 offset:8192\n\t"
      "s_waitcnt vmcnt(16)\n\t"
      "4:\n\t"
      "s_branch 2f\n\t"
      "1:\n\t"
      "s_waitcnt lgkmcnt(0)\n\t"
      "v_mfma_f32_16x16x32_f16 v[0:3], v[160:163], v[224:227], v[0:3]\n\t"
      "ds_read_b128 v[232:235], v254 offset:1024\n\t"
      "v_mfma_f32_16x16x32_f16 v[0:3], v[176:179], v[228:231], v[0:3]\n\t"
      "ds_read_b128 v[236:239], v254 offset:9216\n\t"
      "v_mfma_f32_16x16x32_f16 v[32:35], v[180:183], v[228:231], v[32:35]\n\t"
      "v_mfma_f32_16x16x32_f16 v[32:35], v[164:167], v[224:227], v[32:35]\n\t"
      "v_mfma_f32_16x16x32_f16 v[64:67], v[168:171], v[224:227], v[64:67]\n\t"
      "v_mfma_f32_16x16x32_f16 v[64:67], v[184:187], v[228:231], v[64:67]\n\t"
      "v_mfma_f32_16x16x32_f16 v[96:99], v[188:191], v[228:231], v[96:99]\n\t"
      "v_mfma_f32_16x16x32_f16 v[96:99], v[172:175], v[224:227], v[96:99]\n\t"
      "s_waitcnt lgkmcnt(0)\n\t"
      "v_mfma_f32_16x16x32_f16 v[100:103], v[172:175], v[232:235], v[100:103]\n\t"
      "ds_read_b128 v[224:227], v254 offset:2048\n\t"
      "v_mfma_f32_16x16x32_f16 v[100:103], v[188:191], v[236:239], v[100:103]\n\t"
      "ds_read_b128 v[228:231], v254 offset:10240\n\t"
      "v_mfma_f32_16x16x32_f16 v[68:71], v[184:187], v[236:239], v[68:71]\n\t"
      "v_mfma_f32_16x16x32_f16 v[68:71], v[168:171], v[232:235], v[68:71]\n\t"
      "v_mfma_f32_16x16x32_f16 v[36:39], v[164:167], v[232:235], v[36:39]\n\t"
      "v_mfma_f32_16x16x32_f16 v[36:39], v[180:183], v[236:239], v[36:39]\n\t"
      "v_mfma_f32_16x16x32_f16 v[4:7], v[176:179], v[236:239], v[4:7]\n\t"
      "v_mfma_f32_16x16x32_f16 v[4:7], v[160:163], v[232:235], v[4:7]\n\t"
      "s_waitcnt lgkmcnt(0)\n\t"
      "v_mfma_f32_16x16x32_f16 v[8:11], v[160:163], v[224:227], v[8:11]\n\t"
      "ds_read_b128 v[232:235], v254 offset:3072\n\t"
      "v_mfma_f32_16x16x32_f16 v[8:11], v[176:179], v[228:231], v[8:11]\n\t"
      "ds_read_b128 v[236:239], v254 offset:11264\n\t"
      "v_mfma_f32_16x16x32_f16 v[40:43], v[180:183], v[228:231], v[40:43]\n\t"
      "v_mfma_f32_16x16x32_f16 v[40:43], v[164:167], v[224:227], v[40:43]\n\t"
      "v_mfma_f32_16x16x32_f16 v[72:75], v[168:171], v[224:227], v[72:75]\n\t"
      "v_mfma_f32_16x16x32_f16 v[72:75], v[184:187], v[228:231], v[72:75]\n\t"
      "v_mfma_f32_16x16x32_f16 v[104:107], v[188:191], v[228:231], v[104:107]\n\t"
      "v_mfma_f32_16x16x32_f16 v[104:107], v[172:175], v[224:227], v[104:107]\n\t"
      "s_waitcnt lgkmcnt(0)\n\t"
      "v_mfma_f32_16x16x32_f16 v[108:111], v[172:175], v[232:235], v[108:111]\n\t"
      "ds_read_b128 v[224:227], v254 offset:4096\n\t"
      "v_mfma_f32_16x16x32_f16 v[108:111], v[188:191], v[236:239], v[108:111]\n\t"
      "ds_read_b128 v[228:231], v254 offset:12288\n\t"
      "v_mfma_f32_16x16x32_f16 v[76:79], v[184:187], v[236:239], v[76:79]\n\t"
      "v_mfma_f32_16x16x32_f16 v[76:79], v[168:171], v[232:235], v[76:79]\n\t"
      "v_mfma_f32_16x16x32_f16 v[44:47], v[164:167], v[232:235], v[44:47]\n\t"
      "v_mfma_f32_16x16x32_f16 v[44:47], v[180:183], v[236:239], v[44:47]\n\t"
      "v_mfma_f32_16x16x32_f16 v[12:15], v[176:179], v[236:239], v[12:15]\n\t"
      "v_mfma_f32_16x16x32_f16 v[12:15], v[160:163], v[232:235], v[12:15]\n\t"
      "s_waitcnt lgkmcnt(0)\n\t"
      "v_mfma_f32_16x16x32_f16 v[16:19], v[160:163], v[224:227], v[16:19]\n\t"
      "ds_read_b128 v[232:235], v254 offset:5120\n\t"
      "v_mfma_f32_16x16x32_f16 v[16:19], v[176:179], v[228:231], v[16:19]\n\t"
      "ds_read_b128 v[236:239], v254 offset:13312\n\t"
      "v_mfma_f32_16x16x32_f16 v[48:51], v[180:183], v[228:231], v[48:51]\n\t"
      "v_mfma_f32_16x16x32_f16 v[48:51], v[164:167], v[224:227], v[48:51]\n\t"
      "v_mfma_f32_16x16x32_f16 v[80:83], v[168:171], v[224:227], v[80:83]\n\t"
      "v_mfma_f32_16x16x32_f16 v[80:83], v[184:187], v[228:231], v[80:83]\n\t"
      "v_mfma_f32_16x16x32_f16 v[112:115], v[188:191], v[228:231], v[112:115]\n\t"
      "v_mfma_f32_16x16x32_f16 v[112:115], v[172:175], v[224:227], v[112:115]\n\t"
      "s_waitcnt lgkmcnt(0)\n\t"
      "v_mfma_f32_16x16x32_f16 v[116:119], v[172:175], v[232:235], v[116:119]\n\t"
      "ds_read_b128 v[224:227], v254 offset:6144\n\t"
      "v_mfma_f32_16x16x32_f16 v[116:119], v[188:191], v[236:239], v[116:119]\n\t"
      "ds_read_b128 v[228:231], v254 offset:14336\n\t"
      "v_mfma_f32_16x16x32_f16 v[84:87], v[184:187], v[236:239], v[84:87]\n\t"
      "v_mfma_f32_16x16x32_f16 v[84:87], v[168:171], v[232:235], v[84:87]\n\t"
      "v_mfma_f32_16x16x32_f16 v[52:55], v[164:167], v[232:235], v[52:55]\n\t"
      "v_mfma_f32_16x16x32_f16 v[52:55], v[180:183], v[236:239], v[52:55]\n\t"
      "v_mfma_f32_16x16x32_f16 v[20:23], v[176:179], v[236:239], v[20:23]\n\t"
      "v_mfma_f32_16x16x32_f16 v[20:23], v[160:163], v[232:235], v[20:23]\n\t"
      "s_waitcnt lgkmcnt(0)\n\t"
      "v_mfma_f32_16x16x32_f16 v[24:27], v[160:163], v[224:227], v[24:27]\n\t"
      "ds_read_b128 v[232:235], v254 offset:7168\n\t"
      "v_mfma_f32_16x16x32_f16 v[24:27], v[176:179], v[228:231], v[24:27]\n\t"
      "ds_read_b128 v[236:239], v254 offset:15360\n\t"
      "v_mfma_f32_16x16x32_f16 v[56:59], v[180:183], v[228:231], v[56:59]\n\t"
      "v_mfma_f32_16x16x32_f16 v[56:59], v[164:167], v[224:227], v[56:59]\n\t"
      "v_mfma_f32_16x16x32_f16 v[88:91], v[168:171], v[224:227], v[88:91]\n\t"
      "v_mfma_f32_16x16x32_f16 v[88:91], v[184:187], v[228:231], v[88:91]\n\t"
      "v_mfma_f32_16x16x32_f16 v[120:123], v[188:191], v[228:231], v[120:123]\n\t"
      "v_mfma_f32_16x16x32_f16 v[120:123], v[172:175], v[224:227], v[120:123]\n\t"
      "s_waitcnt lgkmcnt(0)\n\t"
      "v_mfma_f32_16x16x32_f16 v[124:127], v[172:175], v[232:235], v[124:127]\n\t"
      "v_mfma_f32_16x16x32_f16 v[124:127], v[188:191], v[236:239], v[124:127]\n\t"
      "v_mfma_f32_16x16x32_f16 v[92:95], v[184:187], v[236:239], v[92:95]\n\t"
      "v_mfma_f32_16x16x32_f16 v[92:95], v[168:171], v[232:235], v[92:95]\n\t"
      "v_mfma_f32_16x16x32_f16 v[60:63], v[164:167], v[232:235], v[60:63]\n\t"
      "v_mfma_f32_16x16x32_f16 v[60:63], v[180:183], v[236:239], v[60:63]\n\t"
      "v_mfma_f32_16x16x32_f16 v[28:31], v[176:179], v[236:239], v[28:31]\n\t"
      "v_mfma_f32_16x16x32_f16 v[28:31], v[160:163], v[232:235], v[28:31]\n\t"
      "s_barrier\n\t"
      "s_add_u32 s78, s70, 4\n\t"
      "v_readlane_b32 s73, %[tk], s78\n\t"
      "s_nop 3\n\t"
      "s_lshl_b32 s73, s73, 16\n\t"
      "s_add_u32 s73, s73, s80\n\t"
      "s_cmp_lt_u32 s78, 64\n\t"
      "s_cselect_b32 s73, s73, 0x80000000\n\t"
      "s_add_u32 s74, s73, 0x8000\n\t"
      "buffer_load_dwordx4 v[160:163], %[v16], %[srds], s73 offen sc0 sc1\n\t"
      "buffer_load_dwordx4 v[164:167], %[v16], %[srds], s73 offen offset:1024 sc0 sc1\n\t"
      "buffer_load_dwordx4 v[168:171], %[v16], %[srds], s73 offen offset:2048 sc0 sc1\n\t"
      "buffer_load_dwordx4 v[172:175], %[v16], %[srds], s73 offen offset:3072 sc0 sc1\n\t"
      "buffer_load_dwordx4 v[176:179], %[v16], %[srds], s74 offen sc0 sc1\n\t"
      "buffer_load_dwordx4 v[180:183], %[v16], %[srds], s74 offen offset:1024 sc0 sc1\n\t"
      "buffer_load_dwordx4 v[184:187], %[v16], %[srds], s74 offen offset:2048 sc0 sc1\n\t"
      "buffer_load_dwordx4 v[188:191], %[v16], %[srds], s74 offen offset:3072 sc0 sc1\n\t"
      "s_add_u32 s78, s70, 7\n\t"
      "v_readlane_b32 s72, %[tk], s78\n\t"
      "s_nop 3\n\t"
      "s_lshl_b32 s72, s72, 22\n\t"
      "s_cmp_lt_u32 s78, 64\n\t"
      "s_cselect_b32 s72, s72, 0x80000000\n\t"
      "s_waitcnt vmcnt(8)\n\t"
      "v_cmp_le_u32 s[92:93], s79, %[vp]\n\t"
      "s_add_u32 s78, s70, 6\n\t"
      "s_cmp_ge_u32 s78, 64\n\t"
      "s_cbranch_scc1 42f\n\t"
      "s_nop 0\n\t"
      "s_bitcmp1_b64 s[92:93], s78\n\t"
      "s_cbranch_scc1 42f\n\t"
      "s_lshl_b64 s[90:91], 1, s78\n\t"
      "s_cmp_lg_u32 s83, 0\n\t"
      "s_cbranch_scc1 44f\n\t"
      "s_memrealtime s[86:87]\n\t"
      "43:\n\t"
      "v_lshlrev_b32 v255, 10, %[tk]\n\t"
      "s_mov_b64 exec, s[90:91]\n\t"
      "buffer_load_dword %[vp], v255, %[srds], %[fsrc] offen sc0 sc1\n\t"
      "s_mov_b64 exec, -1\n\t"
      "s_memrealtime s[88:89]\n\t"
      "s_waitcnt vmcnt(0) lgkmcnt(0)\n\t"
      "v_cmp_le_u32 s[92:93], s79, %[vp]\n\t"
      "s_sub_u32 s88, s88, s86\n\t"
      "s_nop 3\n\t"
      "s_andn2_b64 s[90:91], s[90:91], s[92:93]\n\t"
      "s_cbranch_scc0 44f\n\t"
      "s_cmp_lt_u32 s88, 10000\n\t"
      "s_cbranch_scc1 46f\n\t"
      "buffer_load_dword v255, off, %[srds], %[tmo] sc1\n\t"
      "s_waitcnt vmcnt(0)\n\t"
      "v_readfirstlane_b32 s77, v255\n\t"
      "s_nop 3\n\t"
      "s_cmp_eq_u32 s77, s79\n\t"
      "s_cbranch_scc1 45f\n\t"
      "s_cmp_gt_u32 s88, 0x2000000\n\t"
      "s_cbranch_scc1 45f\n\t"
      "s_sleep 64\n\t"
      "46:\n\t"
      "s_sleep 4\n\t"
      "s_branch 43b\n\t"
      "45:\n\t"
      "s_mov_b32 s83, 1\n\t"
      "v_mov_b32 v255, s79\n\t"
      "s_mov_b64 exec, 1\n\t"
      "buffer_store_dword v255, off, %[srds], %[tmo] sc1\n\t"
      "s_mov_b64 exec, -1\n\t"
      "44:\n\t"
      "42:\n\t"
      "s_add_u32 s78, s70, 5\n\t"
      "s_cmp_ge_u32 s78, 64\n\t"
      "s_cbranch_scc1 47f\n\t"
      "s_and_b32 s77, s78, 7\n\t"
      "s_lshl_b32 s77, s77, 14\n\t"
      "s_add_u32 s77, s77, %[lw]\n\t"
      "v_add_u32 v255, s77, %[v16]\n\t"
      "v_cvt_pk_f16_f32 v224, v192, v196\n\t"
      "v_cvt_pk_f16_f32 v225, v200, v204\n\t"
      "v_cvt_pk_f16_f32 v226, v208, v212\n\t"
      "v_cvt_pk_f16_f32 v227, v216, v220\n\t"
      "ds_write_b128 v255, v[224:227] offset:0\n\t"
      "v_cvt_pk_f16_f32 v228, v193, v197\n\t"
      "v_cvt_pk_f16_f32 v229, v201, v205\n\t"
      "v_cvt_pk_f16_f32 v230, v209, v213\n\t"
      "v_cvt_pk_f16_f32 v231, v217, v221\n\t"
      "ds_write_b128 v255, v[228:231] offset:1024\n\t"
      "v_cvt_pk_f16_f32 v232, v194, v198\n\t"
      "v_cvt_pk_f16_f32 v233, v202, v206\n\t"
      "v_cvt_pk_f16_f32 v234, v210, v214\n\t"
      "v_cvt_pk_f16_f32 v235, v218, v222\n\t"
      "ds_write_b128 v255, v[232:235] offset:2048\n\t"
      "v_cvt_pk_f16_f32 v236, v195, v199\n\t"
      "v_cvt_pk_f16_f32 v237, v203, v207\n\t"
      "v_cvt_pk_f16_f32 v238, v211, v215\n\t"
      "v_cvt_pk_f16_f32 v239, v219, v223\n\t"
      "ds_write_b128 v255, v[236:239] offset:3072\n\t"
      "buffer_load_dwordx4 v[192:195], %[vw], %[srdw], s72 offen\n\t"
      "s_add_u32 s72, s72, 65536\n\t"
      "buffer_load_dwordx4 v[196:199], %[vw], %[srdw], s72 offen\n\t"
      "s_add_u32 s72, s72, 65536\n\t"
      "buffer_load_dwordx4 v[200:203], %[vw], %[srdw], s72 offen\n\t"
      "s_add_u32 s72, s72, 65536\n\t"
      "buffer_load_dwordx4 v[204:207], %[vw], %[srdw], s72 offen\n\t"
      "s_add_u32 s72, s72, 851968\n\t"
      "buffer_load_dwordx4 v[208:211], %[vw], %[srdw], s72 offen\n\t"
      "s_add_u32 s72, s72, 65536\n\t"
      "buffer_load_dwordx4 v[212:215], %[vw], %[srdw], s72 offen\n\t"
      "s_add_u32 s72, s72, 65536\n\t"
      "buffer_load_dwordx4 v[216:219], %[vw], %[srdw], s72 offen\n\t"
      "s_add_u32 s72, s72, 65536\n\t"
      "buffer_load_dwordx4 v[220:223], %[vw], %[srdw], s72 offen\n\t"
      "47:\n\t"
      "s_add_u32 s78, s70, 8\n\t"
      "s_cmp_ge_u32 s78, 64\n\t"
      "s_cbranch_scc1 48f\n\t"
      "s_mov_b64 s[90:91], 5\n\t"
      "s_lshl_b64 s[90:91], s[90:91], s78\n\t"
      "s_andn2_b64 s[90:91], s[90:91], s[92:93]\n\t"
      "s_cbranch_scc0 48f\n\t"
      "v_lshlrev_b32 v255, 10, %[tk]\n\t"
      "s_mov_b64 exec, s[90:91]\n\t"
      "buffer_load_dword %[vp], v255, %[srds], %[fsrc] offen sc0 sc1\n\t"
      "s_mov_b64 exec, -1\n\t"
      "48:\n\t"
      "s_lshr_b32 s77, s70, 1\n\t"
      "s_xor_b32 s77, s77, 0\n\t"
      "s_and_b32 s77, s77, 15\n\t"
      "s_cmp_lg_u32 s77, %[pj]\n\t"
      "s_cbranch_scc1 49f\n\t"
      "s_add_u32 s78, s70, 6\n\t"
      "s_cmp_ge_u32 s78, 64\n\t"
      "s_cbranch_scc1 49f\n\t"
      "v_readlane_b32 s73, %[tk], s78\n\t"
      "s_nop 3\n\t"
      "s_lshl_b32 s73, s73, 16\n\t"
      "s_add_u32 s73, s73, s80\n\t"
      "s_cmp_lt_u32 s78, 64\n\t"
      "s_cselect_b32 s73, s73, 0x80000000\n\t"
      "s_add_u32 s74, s73, 0x8000\n\t"
      "buffer_load_dword v255, %[vc], %[srds], s73 offen\n\t"
      "49:\n\t"
      "s_sub_u32 s77, s70, s81\n\t"
      "s_cmp_gt_u32 s77, 8\n\t"
      "s_cbranch_scc1 50f\n\t"
      "s_cmp_eq_u32 s77, 2\n\t"
      "s_cbranch_scc1 51f\n\t"
      "s_cmp_eq_u32 s77, 4\n\t"
      "s_cbranch_scc1 52f\n\t"
      "s_cmp_eq_u32 s77, 8\n\t"
      "s_cbranch_scc1 53f\n\t"
      "s_cmp_lg_u32 s77, 0\n\t"
      "s_cbranch_scc1 50f\n\t"
      "buffer_load_dwordx4 v[240:243], %[vx], %[srdx], 0 offen offset:0 nt\n\t"
      "buffer_load_dwordx4 v[244:247], %[vx], %[srdx], 0 offen offset:64 nt\n\t"
      "s_branch 50f\n\t"
      "51:\n\t"
      "v_cvt_pk_f16_f32 v240, v240, v241\n\t"
      "v_cvt_pk_f16_f32 v241, v242, v243\n\t"
      "v_cvt_pk_f16_f32 v242, v244, v245\n\t"
      "v_cvt_pk_f16_f32 v243, v246, v247\n\t"
      "buffer_store_dwordx4 v[240:243], %[v16], %[srdd], s84 offen sc0 sc1\n\t"
      "s_nop 1\n\t"
      "buffer_load_dwordx4 v[240:243], %[vx], %[srdx], 0 offen offset:128 nt\n\t"
      "buffer_load_dwordx4 v[244:247], %[vx], %[srdx], 0 offen offset:192 nt\n\t"
      "s_branch 50f\n\t"
      "52:\n\t"
      "v_cvt_pk_f16_f32 v240, v240, v241\n\t"
      "v_cvt_pk_f16_f32 v241, v242, v243\n\t"
      "v_cvt_pk_f16_f32 v242, v244, v245\n\t"
      "v_cvt_pk_f16_f32 v243, v246, v247\n\t"
      "s_add_u32 s77, s84, 0x8000\n\t"
      "buffer_store_dwordx4 v[240:243], %[v16], %[srdd], s77 offen sc0 sc1\n\t"
      "s_branch 50f\n\t"
      "53:\n\t"
      "s_cmp_eq_u32 %[pub], 0\n\t"
      "s_cbranch_scc1 50f\n\t"
      "v_mov_b32 v240, s79\n\t"
      "s_mov_b64 exec, 1\n\t"
      "buffer_store_dword v240, off, %[srdd], %[fdst] sc0 sc1\n\t"
      "s_mov_b64 exec, -1\n\t"
      "50:\n\t"
      "s_cmp_lt_u32 s85, 190\n\t"
      "s_cbranch_scc1 54f\n\t"
      "55:\n\t"
      "s_memrealtime s[88:89]\n\t"
      "s_waitcnt lgkmcnt(0)\n\t"
      "s_sub_u32 s77, s88, s76\n\t"
      "s_cmp_lt_i32 s77, 0\n\t"
      "s_cbranch_scc1 55b\n\t"
      "s_sub_u32 s77, s88, s75\n\t"
      "s_cmp_gt_u32 s77, 2600\n\t"
      "s_cselect_b32 s78, 3, 0\n\t"
      "s_sub_u32 s85, s85, s78\n\t"
      "s_cmp_gt_u32 s77, 0\n\t"
      "s_cselect_b32 s78, s85, 0\n\t"
      "s_add_u32 s76, s88, s78\n\t"
      "54:\n\t"
      "s_add_u32 s70, s70, 2\n\t"
      "s_and_b32 s77, s70, 7\n\t"
      "s_lshl_b32 s77, s77, 14\n\t"
      "s_add_u32 s77, s77, %[lr]\n\t"
      "v_add_u32 v254, s77, %[v16]\n\t"
      "s_waitcnt lgkmcnt(0)\n\t"
      "ds_read_b128 v[224:227], v254 offset:0\n\t"
      "ds_read_b128 v[228:231], v254 offset:8192\n\t"
      "s_barrier\n\t"
      "2:\n\t"
      "s_waitcnt lgkmcnt(0)\n\t"
      "v_mfma_f32_16x16x32_f16 v[0:3], v[128:131], v[224:227], v[0:3]\n\t"
      "ds_read_b128 v[232:235], v254 offset:1024\n\t"
      "v_mfma_f32_16x16x32_f16 v[0:3], v[144:147], v[228:231], v[0:3]\n\t"
      "ds_read_b128 v[236:239], v254 offset:9216\n\t"
      "v_mfma_f32_16x16x32_f16 v[32:35], v[148:151], v[228:231], v[32:35]\n\t"
      "v_mfma_f32_16x16x32_f16 v[32:35], v[132:135], v[224:227], v[32:35]\n\t"
      "v_mfma_f32_16x16x32_f16 v[64:67], v[136:139], v[224:227], v[64:67]\n\t"
      "v_mfma_f32_16x16x32_f16 v[64:67], v[152:155], v[228:231], v[64:67]\n\t"
      "v_mfma_f32_16x16x32_f16 v[96:99], v[156:159], v[228:231], v[96:99]\n\t"
      "v_mfma_f32_16x16x32_f16 v[96:99], v[140:143], v[224:227], v[96:99]\n\t"
      "s_waitcnt lgkmcnt(0)\n\t"
      "v_mfma_f32_16x16x32_f16 v[100:103], v[140:143], v[232:235], v[100:103]\n\t"
      "ds_read_b128 v[224:227], v254 offset:2048\n\t"
      "v_mfma_f32_16x16x32_f16 v[100:103], v[156:159], v[236:239], v[100:103]\n\t"
      "ds_read_b128 v[228:231], v254 offset:10240\n\t"
      "v_mfma_f32_16x16x32_f16 v[68:71], v[152:155], v[236:239], v[68:71]\n\t"
      "v_mfma_f32_16x16x32_f16 v[68:71], v[136:139], v[232:235], v[68:71]\n\t"
      "v_mfma_f32_16x16x32_f16 v[36:39], v[132:135], v[232:235], v[36:39]\n\t"
      "v_mfma_f32_16x16x32_f16 v[36:39], v[148:151], v[236:239], v[36:39]\n\t"
      "v_mfma_f32_16x16x32_f16 v[4:7], v[144:147], v[236:239], v[4:7]\n\t"
      "v_mfma_f32_16x16x32_f16 v[4:7], v[128:131], v[232:235], v[4:7]\n\t"
      "s_waitcnt lgkmcnt(0)\n\t"
      "v_mfma_f32_16x16x32_f16 v[8:11], v[128:131], v[224:227], v[8:11]\n\t"
      "ds_read_b128 v[232:235], v254 offset:3072\n\t"
      "v_mfma_f32_16x16x32_f16 v[8:11], v[144:147], v[228:231], v[8:11]\n\t"
      "ds_read_b128 v[236:239], v254 offset:11264\n\t"
      "v_mfma_f32_16x16x32_f16 v[40:43], v[148:151], v[228:231], v[40:43]\n\t"
      "v_mfma_f32_16x16x32_f16 v[40:43], v[132:135], v[224:227], v[40:43]\n\t"
      "v_mfma_f32_16x16x32_f16 v[72:75], v[136:139], v[224:227], v[72:75]\n\t"
      "v_mfma_f32_16x16x32_f16 v[72:75], v[152:155], v[228:231], v[72:75]\n\t"
      "v_mfma_f32_16x16x32_f16 v[104:107], v[156:159], v[228:231], v[104:107]\n\t"
      "v_mfma_f32_16x16x32_f16 v[104:107], v[140:143], v[224:227], v[104:107]\n\t"
      "s_waitcnt lgkmcnt(0)\n\t"
      "v_mfma_f32_16x16x32_f16 v[108:111], v[140:143], v[232:235], v[108:111]\n\t"
      "ds_read_b128 v[224:227], v254 offset:4096\n\t"
      "v_mfma_f32_16x16x32_f16 v[108:111], v[156:159], v[236:239], v[108:111]\n\t"
      "ds_read_b128 v[228:231], v254 offset:12288\n\t"
      "v_mfma_f32_16x16x32_f16 v[76:79], v[152:155], v[236:239], v[76:79]\n\t"
      "v_mfma_f32_16x16x32_f16 v[76:79], v[136:139], v[232:235], v[76:79]\n\t"
      "v_mfma_f32_16x16x32_f16 v[44:47], v[132:135], v[232:235], v[44:47]\n\t"
      "v_mfma_f32_16x16x32_f16 v[44:47], v[148:151], v[236:239], v[44:47]\n\t"
      "v_mfma_f32_16x16x32_f16 v[12:15], v[144:147], v[236:239], v[12:15]\n\t"
      "v_mfma_f32_16x16x32_f16 v[12:15], v[128:131], v[232:235], v[12:15]\n\t"
      "s_waitcnt lgkmcnt(0)\n\t"
      "v_mfma_f32_16x16x32_f16 v[16:19], v[128:131], v[224:227], v[16:19]\n\t"
      "ds_read_b128 v[232:235], v254 offset:5120\n\t"
      "v_mfma_f32_16x16x32_f16 v[16:19], v[144:147], v[228:231], v[16:19]\n\t"
      "ds_read_b128 v[236:239], v254 offset:13312\n\t"
      "v_mfma_f32_16x16x32_f16 v[48:51], v[148:151], v[228:231], v[48:51]\n\t"
      "v_mfma_f32_16x16x32_f16 v[48:51], v[132:135], v[224:227], v[48:51]\n\t"
      "v_mfma_f32_16x16x32_f16 v[80:83], v[136:139], v[224:227], v[80:83]\n\t"
      "v_mfma_f32_16x16x32_f16 v[80:83], v[152:155], v[228:231], v[80:83]\n\t"
      "v_mfma_f32_16x16x32_f16 v[112:115], v[156:159], v[228:231], v[112:115]\n\t"
      "v_mfma_f32_16x16x32_f16 v[112:115], v[140:143], v[224:227], v[112:115]\n\t"
      "s_waitcnt lgkmcnt(0)\n\t"
      "v_mfma_f32_16x16x32_f16 v[116:119], v[140:143], v[232:235], v[116:119]\n\t"
      "ds_read_b128 v[224:227], v254 offset:6144\n\t"
      "v_mfma_f32_16x16x32_f16 v[116:119], v[156:159], v[236:239], v[116:119]\n\t"
      "ds_read_b128 v[228:231], v254 offset:14336\n\t"
      "v_mfma_f32_16x16x32_f16 v[84:87], v[152:155], v[236:239], v[84:87]\n\t"
      "v_mfma_f32_16x16x32_f16 v[84:87], v[136:139], v[232:235], v[84:87]\n\t"
      "v_mfma_f32_16x16x32_f16 v[52:55], v[132:135], v[232:235], v[52:55]\n\t"
      "v_mfma_f32_16x16x32_f16 v[52:55], v[148:151], v[236:239], v[52:55]\n\t"
      "v_mfma_f32_16x16x32_f16 v[20:23], v[144:147], v[236:239], v[20:23]\n\t"
      "v_mfma_f32_16x16x32_f16 v[20:23], v[128:131], v[232:235], v[20:23]\n\t"
      "s_waitcnt lgkmcnt(0)\n\t"
      "v_mfma_f32_16x16x32_f16 v[24:27], v[128:131], v[224:227], v[24:27]\n\t"
      "ds_read_b128 v[232:235], v254 offset:7168\n\t"
      "v_mfma_f32_16x16x32_f16 v[24:27], v[144:147], v[228:231], v[24:27]\n\t"
      "ds_read_b128 v[236:239], v254 offset:15360\n\t"
      "v_mfma_f32_16x16x32_f16 v[56:59], v[148:151], v[228:231], v[56:59]\n\t"
      "v_mfma_f32_16x16x32_f16 v[56:59], v[132:135], v[224:227], v[56:59]\n\t"
      "v_mfma_f32_16x16x32_f16 v[88:91], v[136:139], v[224:227], v[88:91]\n\t"
      "v_mfma_f32_16x16x32_f16 v[88:91], v[152:155], v[228:231], v[88:91]\n\t"
      "v_mfma_f32_16x16x32_f16 v[120:123], v[156:159], v[228:231], v[120:123]\n\t"
      "v_mfma_f32_16x16x32_f16 v[120:123], v[140:143], v[224:227], v[120:123]\n\t"
      "s_waitcnt lgkmcnt(0)\n\t"
      "v_mfma_f32_16x16x32_f16 v[124:127], v[140:143], v[232:235], v[124:127]\n\t"
      "v_mfma_f32_16x16x32_f16 v[124:127], v[156:159], v[236:239], v[124:127]\n\t"
      "v_mfma_f32_16x16x32_f16 v[92:95], v[152:155], v[236:239], v[92:95]\n\t"
      "v_mfma_f32_16x16x32_f16 v[92:95], v[136:139], v[232:235], v[92:95]\n\t"
      "v_mfma_f32_16x16x32_f16 v[60:63], v[132:135], v[232:235], v[60:63]\n\t"
      "v_mfma_f32_16x16x32_f16 v[60:63], v[148:151], v[236:239], v[60:63]\n\t"
      "v_mfma_f32_16x16x32_f16 v[28:31], v[144:147], v[236:239], v[28:31]\n\t"
      "v_mfma_f32_16x16x32_f16 v[28:31], v[128:131], v[232:235], v[28:31]\n\t"
      "s_barrier\n\t"
      "s_add_u32 s78, s70, 4\n\t"
      "v_readlane_b32 s73, %[tk], s78\n\t"
      "s_nop 3\n\t"
      "s_lshl_b32 s73, s73, 16\n\t"
      "s_add_u32 s73, s73, s80\n\t"
      "s_cmp_lt_u32 s78, 64\n\t"
      "s_cselect_b32 s73, s73, 0x80000000\n\t"
      "s_add_u32 s74, s73, 0x8000\n\t"
      "buffer_load_dwordx4 v[128:131], %[v16], %[srds], s73 offen sc0 sc1\n\t"
      "buffer_load_dwordx4 v[132:135], %[v16], %[srds], s73 offen offset:1024 sc0 sc1\n\t"
      "buffer_load_dwordx4 v[136:139], %[v16], %[srds], s73 offen offset:2048 sc0 sc1\n\t"
      "buffer_load_dwordx4 v[140:143], %[v16], %[srds], s73 offen offset:3072 sc0 sc1\n\t"
      "buffer_load_dwordx4 v[144:147], %[v16], %[srds], s74 offen sc0 sc1\n\t"
      "buffer_load_dwordx4 v[148:151], %[v16], %[srds], s74 offen offset:1024 sc0 sc1\n\t"
      "buffer_load_dwordx4 v[152:155], %[v16], %[srds], s74 offen offset:2048 sc0 sc1\n\t"
      "buffer_load_dwordx4 v[156:159], %[v16], %[srds], s74 offen offset:3072 sc0 sc1\n\t"
      "s_add_u32 s78, s70, 7\n\t"
      "v_readlane_b32 s72, %[tk], s78\n\t"
      "s_nop 3\n\t"
      "s_lshl_b32 s72, s72, 22\n\t"
      "s_cmp_lt_u32 s78, 64\n\t"
      "s_cselect_b32 s72, s72, 0x80000000\n\t"
      "s_waitcnt vmcnt(8)\n\t"
      "v_cmp_le_u32 s[92:93], s79, %[vp]\n\t"
      "s_add_u32 s78, s70, 6\n\t"
      "s_cmp_ge_u32 s78, 64\n\t"
      "s_cbranch_scc1 56f\n\t"
      "s_nop 0\n\t"
      "s_bitcmp1_b64 s[92:93], s78\n\t"
      "s_cbranch_scc1 56f\n\t"
      "s_lshl_b64 s[90:91], 1, s78\n\t"
      "s_cmp_lg_u32 s83, 0\n\t"
      "s_cbranch_scc1 58f\n\t"
      "s_memrealtime s[86:87]\n\t"
      "57:\n\t"
      "v_lshlrev_b32 v255, 10, %[tk]\n\t"
      "s_mov_b64 exec, s[90:91]\n\t"
      "buffer_load_dword %[vp], v255, %[srds], %[fsrc] offen sc0 sc1\n\t"
      "s_mov_b64 exec, -1\n\t"
      "s_memrealtime s[88:89]\n\t"
      "s_waitcnt vmcnt(0) lgkmcnt(0)\n\t"
      "v_cmp_le_u32 s[92:93], s79, %[vp]\n\t"
      "s_sub_u32 s88, s88, s86\n\t"
      "s_nop 3\n\t"
      "s_andn2_b64 s[90:91], s[90:91], s[92:93]\n\t"
      "s_cbranch_scc0 58f\n\t"
      "s_cmp_lt_u32 s88, 10000\n\t"
      "s_cbranch_scc1 60f\n\t"
      "buffer_load_dword v255, off, %[srds], %[tmo] sc1\n\t"
      "s_waitcnt vmcnt(0)\n\t"
      "v_readfirstlane_b32 s77, v255\n\t"
      "s_nop 3\n\t"
      "s_cmp_eq_u32 s77, s79\n\t"
      "s_cbranch_scc1 59f\n\t"
      "s_cmp_gt_u32 s88, 0x2000000\n\t"
      "s_cbranch_scc1 59f\n\t"
      "s_sleep 64\n\t"
      "60:\n\t"
      "s_sleep 4\n\t"
      "s_branch 57b\n\t"
      "59:\n\t"
      "s_mov_b32 s83, 1\n\t"
      "v_mov_b32 v255, s79\n\t"
      "s_mov_b64 exec, 1\n\t"
      "buffer_store_dword v255, off, %[srds], %[tmo] sc1\n\t"
      "s_mov_b64 exec, -1\n\t"
      "58:\n\t"
      "56:\n\t"
      "s_add_u32 s78, s70, 5\n\t"
      "s_cmp_ge_u32 s78, 64\n\t"
      "s_cbranch_scc1 61f\n\t"
      "s_and_b32 s77, s78, 7\n\t"
      "s_lshl_b32 s77, s77, 14\n\t"
      "s_add_u32 s77, s77, %[lw]\n\t"
      "v_add_u32 v255, s77, %[v16]\n\t"
      "v_cvt_pk_f16_f32 v224, v192, v196\n\t"
      "v_cvt_pk_f16_f32 v225, v200, v204\n\t"
      "v_cvt_pk_f16_f32 v226, v208, v212\n\t"
      "v_cvt_pk_f16_f32 v227, v216, v220\n\t"
      "ds_write_b128 v255, v[224:227] offset:0\n\t"
      "v_cvt_pk_f16_f32 v228, v193, v197\n\t"
      "v_cvt_pk_f16_f32 v229, v201, v205\n\t"
      "v_cvt_pk_f16_f32 v230, v209, v213\n\t"
      "v_cvt_pk_f16_f32 v231, v217, v221\n\t"
      "ds_write_b128 v255, v[228:231] offset:1024\n\t"
      "v_cvt_pk_f16_f32 v232, v194, v198\n\t"
      "v_cvt_pk_f16_f32 v233, v202, v206\n\t"
      "v_cvt_pk_f16_f32 v234, v210, v214\n\t"
      "v_cvt_pk_f16_f32 v235, v218, v222\n\t"
      "ds_write_b128 v255, v[232:235] offset:2048\n\t"
      "v_cvt_pk_f16_f32 v236, v195, v199\n\t"
      "v_cvt_pk_f16_f32 v237, v203, v207\n\t"
      "v_cvt_pk_f16_f32 v238, v211, v215\n\t"
      "v_cvt_pk_f16_f32 v239, v219, v223\n\t"
      "ds_write_b128 v255, v[236:239] offset:3072\n\t"
      "buffer_load_dwordx4 v[192:195], %[vw], %[srdw], s72 offen\n\t"
      "s_add_u32 s72, s72, 65536\n\t"
      "buffer_load_dwordx4 v[196:199], %[vw], %[srdw], s72 offen\n\t"
      "s_add_u32 s72, s72, 65536\n\t"
      "buffer_load_dwordx4 v[200:203], %[vw], %[srdw], s72 offen\n\t"
      "s_add_u32 s72, s72, 65536\n\t"
      "buffer_load_dwordx4 v[204:207], %[vw], %[srdw], s72 offen\n\t"
      "s_add_u32 s72, s72, 851968\n\t"
      "buffer_load_dwordx4 v[208:211], %[vw], %[srdw], s72 offen\n\t"
      "s_add_u32 s72, s72, 65536\n\t"
      "buffer_load_dwordx4 v[212:215], %[vw], %[srdw], s72 offen\n\t"
      "s_add_u32 s72, s72, 65536\n\t"
      "buffer_load_dwordx4 v[216:219], %[vw], %[srdw], s72 offen\n\t"
      "s_add_u32 s72, s72, 65536\n\t"
      "buffer_load_dwordx4 v[220:223], %[vw], %[srdw], s72 offen\n\t"
      "61:\n\t"
      "s_add_u32 s78, s70, 8\n\t"
      "s_cmp_ge_u32 s78, 64\n\t"
      "s_cbranch_scc1 62f\n\t"
      "s_mov_b64 s[90:91], 5\n\t"
      "s_lshl_b64 s[90:91], s[90:91], s78\n\t"
      "s_andn2_b64 s[90:91], s[90:91], s[92:93]\n\t"
      "s_cbranch_scc0 62f\n\t"
      "v_lshlrev_b32 v255, 10, %[tk]\n\t"
      "s_mov_b64 exec, s[90:91]\n\t"
      "buffer_load_dword %[vp], v255, %[srds], %[fsrc] offen sc0 sc1\n\t"
      "s_mov_b64 exec, -1\n\t"
      "62:\n\t"
      "s_lshr_b32 s77, s70, 1\n\t"
      "s_xor_b32 s77, s77, 0\n\t"
      "s_and_b32 s77, s77, 15\n\t"
      "s_cmp_lg_u32 s77, %[pj]\n\t"
      "s_cbranch_scc1 63f\n\t"
      "s_add_u32 s78, s70, 6\n\t"
      "s_cmp_ge_u32 s78, 64\n\t"
      "s_cbranch_scc1 63f\n\t"
      "v_readlane_b32 s73, %[tk], s78\n\t"
      "s_nop 3\n\t"
      "s_lshl_b32 s73, s73, 16\n\t"
      "s_add_u32 s73, s73, s80\n\t"
      "s_cmp_lt_u32 s78, 64\n\t"
      "s_cselect_b32 s73, s73, 0x80000000\n\t"
      "s_add_u32 s74, s73, 0x8000\n\t"
      "buffer_load_dword v255, %[vc], %[srds], s73 offen\n\t"
      "63:\n\t"
      "s_sub_u32 s77, s70, s81\n\t"
      "s_cmp_gt_u32 s77, 8\n\t"
      "s_cbranch_scc1 64f\n\t"
      "s_cmp_eq_u32 s77, 2\n\t"
      "s_cbranch_scc1 65f\n\t"
      "s_cmp_eq_u32 s77, 4\n\t"
      "s_cbranch_scc1 66f\n\t"
      "s_cmp_eq_u32 s77, 8\n\t"
      "s_cbranch_scc1 67f\n\t"
      "s_cmp_lg_u32 s77, 0\n\t"
      "s_cbranch_scc1 64f\n\t"
      "buffer_load_dwordx4 v[240:243], %[vx], %[srdx], 0 offen offset:0 nt\n\t"
      "buffer_load_dwordx4 v[244:247], %[vx], %[srdx], 0 offen offset:64 nt\n\t"
      "s_branch 64f\n\t"
      "65:\n\t"
      "v_cvt_pk_f16_f32 v240, v240, v241\n\t"
      "v_cvt_pk_f16_f32 v241, v242, v243\n\t"
      "v_cvt_pk_f16_f32 v242, v244, v245\n\t"
      "v_cvt_pk_f16_f32 v243, v246, v247\n\t"
      "buffer_store_dwordx4 v[240:243], %[v16], %[srdd], s84 offen sc0 sc1\n\t"
      "s_nop 1\n\t"
      "buffer_load_dwordx4 v[240:243], %[vx], %[srdx], 0 offen offset:128 nt\n\t"
      "buffer_load_dwordx4 v[244:247], %[vx], %[srdx], 0 offen offset:192 nt\n\t"
      "s_branch 64f\n\t"
      "66:\n\t"
      "v_cvt_pk_f16_f32 v240, v240, v241\n\t"
      "v_cvt_pk_f16_f32 v241, v242, v243\n\t"
      "v_cvt_pk_f16_f32 v242, v244, v245\n\t"
      "v_cvt_pk_f16_f32 v243, v246, v247\n\t"
      "s_add_u32 s77, s84, 0x8000\n\t"
      "buffer_store_dwordx4 v[240:243], %[v16], %[srdd], s77 offen sc0 sc1\n\t"
      "s_branch 64f\n\t"
      "67:\n\t"
      "s_cmp_eq_u32 %[pub], 0\n\t"
      "s_cbranch_scc1 64f\n\t"
      "v_mov_b32 v240, s79\n\t"
      "s_mov_b64 exec, 1\n\t"
      "buffer_store_dword v240, off, %[srdd], %[fdst] sc0 sc1\n\t"
      "s_mov_b64 exec, -1\n\t"
      "64:\n\t"
      "s_cmp_lt_u32 s85, 190\n\t"
      "s_cbranch_scc1 68f\n\t"
      "69:\n\t"
      "s_memrealtime s[88:89]\n\t"
      "s_waitcnt lgkmcnt(0)\n\t"
      "s_sub_u32 s77, s88, s76\n\t"
      "s_cmp_lt_i32 s77, 0\n\t"
      "s_cbranch_scc1 69b\n\t"
      "s_sub_u32 s77, s88, s75\n\t"
      "s_cmp_gt_u32 s77, 2600\n\t"
      "s_cselect_b32 s78, 3, 0\n\t"
      "s_sub_u32 s85, s85, s78\n\t"
      "s_cmp_gt_u32 s77, 0\n\t"
      "s_cselect_b32 s78, s85, 0\n\t"
      "s_add_u32 s76, s88, s78\n\t"
      "68:\n\t"
      "s_add_u32 s70, s70, 2\n\t"
      "s_and_b32 s77, s70, 7\n\t"
      "s_lshl_b32 s77, s77, 14\n\t"
      "s_add_u32 s77, s77, %[lr]\n\t"
      "v_add_u32 v254, s77, %[v16]\n\t"
      "s_waitcnt lgkmcnt(0)\n\t"
      "ds_read_b128 v[224:227], v254 offset:0\n\t"
      "ds_read_b128 v[228:231], v254 offset:8192\n\t"
      "s_barrier\n\t"
      "s_cmp_lt_u32 s70, 62\n\t"
      "s_cbranch_scc1 1b\n\t"
      "s_waitcnt lgkmcnt(0)\n\t"
      "v_mfma_f32_16x16x32_f16 v[0:3], v[160:163], v[224:227], v[0:3]\n\t"
      "ds_read_b128 v[232:235], v254 offset:1024\n\t"
      "v_mfma_f32_16x16x32_f16 v[0:3], v[176:179], v[228:231], v[0:3]\n\t"
      "ds_read_b128 v[236:239], v254 offset:9216\n\t"
      "v_mfma_f32_16x16x32_f16 v[32:35], v[180:183], v[228:231], v[32:35]\n\t"
      "v_mfma_f32_16x16x32_f16 v[32:35], v[164:167], v[224:227], v[32:35]\n\t"
      "v_mfma_f32_16x16x32_f16 v[64:67], v[168:171], v[224:227], v[64:67]\n\t"
      "v_mfma_f32_16x16x32_f16 v[64:67], v[184:187], v[228:231], v[64:67]\n\t"
      "v_mfma_f32_16x16x32_f16 v[96:99], v[188:191], v[228:231], v[96:99]\n\t"
      "v_mfma_f32_16x16x32_f16 v[96:99], v[172:175], v[224:227], v[96:99]\n\t"
      "s_waitcnt lgkmcnt(0)\n\t"
      "v_mfma_f32_16x16x32_f16 v[100:103], v[172:175], v[232:235], v[100:103]\n\t"
      "ds_read_b128 v[224:227], v254 offset:2048\n\t"
      "v_mfma_f32_16x16x32_f16 v[100:103], v[188:191], v[236:239], v[100:103]\n\t"
      "ds_read_b128 v[228:231], v254 offset:10240\n\t"
      "v_mfma_f32_16x16x32_f16 v[68:71], v[184:187], v[236:239], v[68:71]\n\t"
      "v_mfma_f32_16x16x32_f16 v[68:71], v[168:171], v[232:235], v[68:71]\n\t"
      "v_mfma_f32_16x16x32_f16 v[36:39], v[164:167], v[232:235], v[36:39]\n\t"
      "v_mfma_f32_16x16x32_f16 v[36:39], v[180:183], v[236:239], v[36:39]\n\t"
      "v_mfma_f32_16x16x32_f16 v[4:7], v[176:179], v[236:239], v[4:7]\n\t"
      "v_mfma_f32_16x16x32_f16 v[4:7], v[160:163], v[232:235], v[4:7]\n\t"
      "s_waitcnt lgkmcnt(0)\n\t"
      "v_mfma_f32_16x16x32_f16 v[8:11], v[160:163], v[224:227], v[8:11]\n\t"
      "ds_read_b128 v[232:235], v254 offset:3072\n\t"
      "v_mfma_f32_16x16x32_f16 v[8:11], v[176:179], v[228:231], v[8:11]\n\t"
      "ds_read_b128 v[236:239], v254 offset:11264\n\t"
      "v_mfma_f32_16x16x32_f16 v[40:43], v[180:183], v[228:231], v[40:43]\n\t"
      "v_mfma_f32_16x16x32_f16 v[40:43], v[164:167], v[224:227], v[40:43]\n\t"
      "v_mfma_f32_16x16x32_f16 v[72:75], v[168:171], v[224:227], v[72:75]\n\t"
      "v_mfma_f32_16x16x32_f16 v[72:75], v[184:187], v[228:231], v[72:75]\n\t"
      "v_mfma_f32_16x16x32_f16 v[104:107], v[188:191], v[228:231], v[104:107]\n\t"
      "v_mfma_f32_16x16x32_f16 v[104:107], v[172:175], v[224:227], v[104:107]\n\t"
      "s_waitcnt lgkmcnt(0)\n\t"
      "v_mfma_f32_16x16x32_f16 v[108:111], v[172:175], v[232:235], v[108:111]\n\t"
      "ds_read_b128 v[224:227], v254 offset:4096\n\t"
      "v_mfma_f32_16x16x32_f16 v[108:111], v[188:191], v[236:239], v[108:111]\n\t"
      "ds_read_b128 v[228:231], v254 offset:12288\n\t"
      "v_mfma_f32_16x16x32_f16 v[76:79], v[184:187], v[236:239], v[76:79]\n\t"
      "v_mfma_f32_16x16x32_f16 v[76:79], v[168:171], v[232:235], v[76:79]\n\t"
      "v_mfma_f32_16x16x32_f16 v[44:47], v[164:167], v[232:235], v[44:47]\n\t"
      "v_mfma_f32_16x16x32_f16 v[44:47], v[180:183], v[236:239], v[44:47]\n\t"
      "v_mfma_f32_16x16x32_f16 v[12:15], v[176:179], v[236:239], v[12:15]\n\t"
      "v_mfma_f32_16x16x32_f16 v[12:15], v[160:163], v[232:235], v[12:15]\n\t"
      "s_waitcnt lgkmcnt(0)\n\t"
      "v_mfma_f32_16x16x32_f16 v[16:19], v[160:163], v[224:227], v[16:19]\n\t"
      "ds_read_b128 v[232:235], v254 offset:5120\n\t"
      "v_mfma_f32_16x16x32_f16 v[16:19], v[176:179], v[228:231], v[16:19]\n\t"
      "ds_read_b128 v[236:239], v254 offset:13312\n\t"
      "v_mfma_f32_16x16x32_f16 v[48:51], v[180:183], v[228:231], v[48:51]\n\t"
      "v_mfma_f32_16x16x32_f16 v[48:51], v[164:167], v[224:227], v[48:51]\n\t"
      "v_mfma_f32_16x16x32_f16 v[80:83], v[168:171], v[224:227], v[80:83]\n\t"
      "v_mfma_f32_16x16x32_f16 v[80:83], v[184:187], v[228:231], v[80:83]\n\t"
      "v_mfma_f32_16x16x32_f16 v[112:115], v[188:191], v[228:231], v[112:115]\n\t"
      "v_mfma_f32_16x16x32_f16 v[112:115], v[172:175], v[224:227], v[112:115]\n\t"
      "s_waitcnt lgkmcnt(0)\n\t"
      "v_mfma_f32_16x16x32_f16 v[116:119], v[172:175], v[232:235], v[116:119]\n\t"
      "ds_read_b128 v[224:227], v254 offset:6144\n\t"
      "v_mfma_f32_16x16x32_f16 v[116:119], v[188:191], v[236:239], v[116:119]\n\t"
      "ds_read_b128 v[228:231], v254 offset:14336\n\t"
      "v_mfma_f32_16x16x32_f16 v[84:87], v[184:187], v[236:239], v[84:87]\n\t"
      "v_mfma_f32_16x16x32_f16 v[84:87], v[168:171], v[232:235], v[84:87]\n\t"
      "v_mfma_f32_16x16x32_f16 v[52:55], v[164:167], v[232:235], v[52:55]\n\t"
      "v_mfma_f32_16x16x32_f16 v[52:55], v[180:183], v[236:239], v[52:55]\n\t"
      "v_mfma_f32_16x16x32_f16 v[20:23], v[176:179], v[236:239], v[20:23]\n\t"
      "v_mfma_f32_16x16x32_f16 v[20:23], v[160:163], v[232:235], v[20:23]\n\t"
      "s_waitcnt lgkmcnt(0)\n\t"
      "v_mfma_f32_16x16x32_f16 v[24:27], v[160:163], v[224:227], v[24:27]\n\t"
      "ds_read_b128 v[232:235], v254 offset:7168\n\t"
      "v_mfma_f32_16x16x32_f16 v[24:27], v[176:179], v[228:231], v[24:27]\n\t"
      "ds_read_b128 v[236:239], v254 offset:15360\n\t"
      "v_mfma_f32_16x16x32_f16 v[56:59], v[180:183], v[228:231], v[56:59]\n\t"
      "v_mfma_f32_16x16x32_f16 v[56:59], v[164:167], v[224:227], v[56:59]\n\t"
      "v_mfma_f32_16x16x32_f16 v[88:91], v[168:171], v[224:227], v[88:91]\n\t"
      "v_mfma_f32_16x16x32_f16 v[88:91], v[184:187], v[228:231], v[88:91]\n\t"
      "v_mfma_f32_16x16x32_f16 v[120:123], v[188:191], v[228:231], v[120:123]\n\t"
      "v_mfma_f32_16x16x32_f16 v[120:123], v[172:175], v[224:227], v[120:123]\n\t"
      "s_waitcnt lgkmcnt(0)\n\t"
      "v_mfma_f32_16x16x32_f16 v[124:127], v[172:175], v[232:235], v[124:127]\n\t"
      "v_mfma_f32_16x16x32_f16 v[124:127], v[188:191], v[236:239], v[124:127]\n\t"
      "v_mfma_f32_16x16x32_f16 v[92:95], v[184:187], v[236:239], v[92:95]\n\t"
      "v_mfma_f32_16x16x32_f16 v[92:95], v[168:171], v[232:235], v[92:95]\n\t"
      "v_mfma_f32_16x16x32_f16 v[60:63], v[164:167], v[232:235], v[60:63]\n\t"
      "v_mfma_f32_16x16x32_f16 v[60:63], v[180:183], v[236:239], v[60:63]\n\t"
      "v_mfma_f32_16x16x32_f16 v[28:31], v[176:179], v[236:239], v[28:31]\n\t"
      "v_mfma_f32_16x16x32_f16 v[28:31], v[160:163], v[232:235], v[28:31]\n\t"
      "s_cmp_eq_u32 s83, 0\n\t"
      "s_cbranch_scc1 70f\n\t"
      "s_nop 7\n\t"
      "v_mov_b32 v0, 0x7fc00000\n\t"
      "v_mov_b32 v1, 0x7fc00000\n\t"
      "v_mov_b32 v2, 0x7fc00000\n\t"
      "v_mov_b32 v3, 0x7fc00000\n\t"
      "v_mov_b32 v4, 0x7fc00000\n\t"
      "v_mov_b32 v5, 0x7fc00000\n\t"
      "v_mov_b32 v6, 0x7fc00000\n\t"
      "v_mov_b32 v7, 0x7fc00000\n\t"
      "v_mov_b32 v8, 0x7fc00000\n\t"
      "v_mov_b32 v9, 0x7fc00000\n\t"
      "v_mov_b32 v10, 0x7fc00000\n\t"
      "v_mov_b32 v11, 0x7fc00000\n\t"
      "v_mov_b32 v12, 0x7fc00000\n\t"
      "v_mov_b32 v13, 0x7fc00000\n\t"
      "v_mov_b32 v14, 0x7fc00000\n\t"
      "v_mov_b32 v15, 0x7fc00000\n\t"
      "v_mov_b32 v16, 0x7fc00000\n\t"
      "v_mov_b32 v17, 0x7fc00000\n\t"
      "v_mov_b32 v18, 0x7fc00000\n\t"
      "v_mov_b32 v19, 0x7fc00000\n\t"
      "v_mov_b32 v20, 0x7fc00000\n\t"
      "v_mov_b32 v21, 0x7fc00000\n\t"
      "v_mov_b32 v22, 0x7fc00000\n\t"
      "v_mov_b32 v23, 0x7fc00000\n\t"
      "v_mov_b32 v24, 0x7fc00000\n\t"
      "v_mov_b32 v25, 0x7fc00000\n\t"
      "v_mov_b32 v26, 0x7fc00000\n\t"
      "v_mov_b32 v27, 0x7fc00000\n\t"
      "v_mov_b32 v28, 0x7fc00000\n\t"
      "v_mov_b32 v29, 0x7fc00000\n\t"
      "v_mov_b32 v30, 0x7fc00000\n\t"
      "v_mov_b32 v31, 0x7fc00000\n\t"
      "v_mov_b32 v32, 0x7fc00000\n\t"
      "v_mov_b32 v33, 0x7fc00000\n\t"
      "v_mov_b32 v34, 0x7fc00000\n\t"
      "v_mov_b32 v35, 0x7fc00000\n\t"
      "v_mov_b32 v36, 0x7fc00000\n\t"
      "v_mov_b32 v37, 0x7fc00000\n\t"
      "v_mov_b32 v38, 0x7fc00000\n\t"
      "v_mov_b32 v39, 0x7fc00000\n\t"
      "v_mov_b32 v40, 0x7fc00000\n\t"
      "v_mov_b32 v41, 0x7fc00000\n\t"
      "v_mov_b32 v42, 0x7fc00000\n\t"
      "v_mov_b32 v43, 0x7fc00000\n\t"
      "v_mov_b32 v44, 0x7fc00000\n\t"
      "v_mov_b32 v45, 0x7fc00000\n\t"
      "v_mov_b32 v46, 0x7fc00000\n\t"
      "v_mov_b32 v47, 0x7fc00000\n\t"
      "v_mov_b32 v48, 0x7fc00000\n\t"
      "v_mov_b32 v49, 0x7fc00000\n\t"
      "v_mov_b32 v50, 0x7fc00000\n\t"
      "v_mov_b32 v51, 0x7fc00000\n\t"
      "v_mov_b32 v52, 0x7fc00000\n\t"
      "v_mov_b32 v53, 0x7fc00000\n\t"
      "v_mov_b32 v54, 0x7fc00000\n\t"
      "v_mov_b32 v55, 0x7fc00000\n\t"
      "v_mov_b32 v56, 0x7fc00000\n\t"
      "v_mov_b32 v57, 0x7fc00000\n\t"
      "v_mov_b32 v58, 0x7fc00000\n\t"
      "v_mov_b32 v59, 0x7fc00000\n\t"
      "v_mov_b32 v60, 0x7fc00000\n\t"
      "v_mov_b32 v61, 0x7fc00000\n\t"
      "v_mov_b32 v62, 0x7fc00000\n\t"
      "v_mov_b32 v63, 0x7fc00000\n\t"
      "v_mov_b32 v64, 0x7fc00000\n\t"
      "v_mov_b32 v65, 0x7fc00000\n\t"
      "v_mov_b32 v66, 0x7fc00000\n\t"
      "v_mov_b32 v67, 0x7fc00000\n\t"
      "v_mov_b32 v68, 0x7fc00000\n\t"
      "v_mov_b32 v69, 0x7fc00000\n\t"
      "v_mov_b32 v70, 0x7fc00000\n\t"
      "v_mov_b32 v71, 0x7fc00000\n\t"
      "v_mov_b32 v72, 0x7fc00000\n\t"
      "v_mov_b32 v73, 0x7fc00000\n\t"
      "v_mov_b32 v74, 0x7fc00000\n\t"
      "v_mov_b32 v75, 0x7fc00000\n\t"
      "v_mov_b32 v76, 0x7fc00000\n\t"
      "v_mov_b32 v77, 0x7fc00000\n\t"
      "v_mov_b32 v78, 0x7fc00000\n\t"
      "v_mov_b32 v79, 0x7fc00000\n\t"
      "v_mov_b32 v80, 0x7fc00000\n\t"
      "v_mov_b32 v81, 0x7fc00000\n\t"
      "v_mov_b32 v82, 0x7fc00000\n\t"
      "v_mov_b32 v83, 0x7fc00000\n\t"
      "v_mov_b32 v84, 0x7fc00000\n\t"
      "v_mov_b32 v85, 0x7fc00000\n\t"
      "v_mov_b32 v86, 0x7fc00000\n\t"
      "v_mov_b32 v87, 0x7fc00000\n\t"
      "v_mov_b32 v88, 0x7fc00000\n\t"
      "v_mov_b32 v89, 0x7fc00000\n\t"
      "v_mov_b32 v90, 0x7fc00000\n\t"
      "v_mov_b32 v91, 0x7fc00000\n\t"
      "v_mov_b32 v92, 0x7fc00000\n\t"
      "v_mov_b32 v93, 0x7fc00000\n\t"
      "v_mov_b32 v94, 0x7fc00000\n\t"
      "v_mov_b32 v95, 0x7fc00000\n\t"
      "v_mov_b32 v96, 0x7fc00000\n\t"
      "v_mov_b32 v97, 0x7fc00000\n\t"
      "v_mov_b32 v98, 0x7fc00000\n\t"
      "v_mov_b32 v99, 0x7fc00000\n\t"
      "v_mov_b32 v100, 0x7fc00000\n\t"
      "v_mov_b32 v101, 0x7fc00000\n\t"
      "v_mov_b32 v102, 0x7fc00000\n\t"
      "v_mov_b32 v103, 0x7fc00000\n\t"
      "v_mov_b32 v104, 0x7fc00000\n\t"
      "v_mov_b32 v105, 0x7fc00000\n\t"
      "v_mov_b32 v106, 0x7fc00000\n\t"
      "v_mov_b32 v107, 0x7fc00000\n\t"
      "v_mov_b32 v108, 0x7fc00000\n\t"
      "v_mov_b32 v109, 0x7fc00000\n\t"
      "v_mov_b32 v110, 0x7fc00000\n\t"
      "v_mov_b32 v111, 0x7fc00000\n\t"
      "v_mov_b32 v112, 0x7fc00000\n\t"
      "v_mov_b32 v113, 0x7fc00000\n\t"
      "v_mov_b32 v114, 0x7fc00000\n\t"
      "v_mov_b32 v115, 0x7fc00000\n\t"
      "v_mov_b32 v116, 0x7fc00000\n\t"
      "v_mov_b32 v117, 0x7fc00000\n\t"
      "v_mov_b32 v118, 0x7fc00000\n\t"
      "v_mov_b32 v119, 0x7fc00000\n\t"
      "v_mov_b32 v120, 0x7fc00000\n\t"
      "v_mov_b32 v121, 0x7fc00000\n\t"
      "v_mov_b32 v122, 0x7fc00000\n\t"
      "v_mov_b32 v123, 0x7fc00000\n\t"
      "v_mov_b32 v124, 0x7fc00000\n\t"
      "v_mov_b32 v125, 0x7fc00000\n\t"
      "v_mov_b32 v126, 0x7fc00000\n\t"
      "v_mov_b32 v127, 0x7fc00000\n\t"
      "70:\n\t"
      "s_cmp_lg_u32 %[grp], 0\n\t"
      "s_cbranch_scc1 5f\n\t"
      "s_barrier\n\t"
      "s_nop 7\n\t"
      "v_add_u32 v254, %[lx], %[v16]\n\t"
      "ds_write_b128 v254, v[0:3] offset:0\n\t"
      "ds_write_b128 v254, v[4:7] offset:1024\n\t"
      "ds_write_b128 v254, v[8:11] offset:2048\n\t"
      "ds_write_b128 v254, v[12:15] offset:3072\n\t"
      "ds_write_b128 v254, v[16:19] offset:4096\n\t"
      "ds_write_b128 v254, v[20:23] offset:5120\n\t"
      "ds_write_b128 v254, v[24:27] offset:6144\n\t"
      "ds_write_b128 v254, v[28:31] offset:7168\n\t"
      "ds_write_b128 v254, v[32:35] offset:8192\n\t"
      "ds_write_b128 v254, v[36:39] offset:9216\n\t"
      "ds_write_b128 v254, v[40:43] offset:10240\n\t"
      "ds_write_b128 v254, v[44:47] offset:11264\n\t"
      "ds_write_b128 v254, v[48:51] offset:12288\n\t"
      "ds_write_b128 v254, v[52:55] offset:13312\n\t"
      "ds_write_b128 v254, v[56:59] offset:14336\n\t"
      "ds_write_b128 v254, v[60:63] offset:15360\n\t"
      "ds_write_b128 v254, v[64:67] offset:16384\n\t"
      "ds_write_b128 v254, v[68:71] offset:17408\n\t"
      "ds_write_b128 v254, v[72:75] offset:18432\n\t"
      "ds_write_b128 v254, v[76:79] offset:19456\n\t"
      "ds_write_b128 v254, v[80:83] offset:20480\n\t"
      "ds_write_b128 v254, v[84:87] offset:21504\n\t"
      "ds_write_b128 v254, v[88:91] offset:22528\n\t"
      "ds_write_b128 v254, v[92:95] offset:23552\n\t"
      "ds_write_b128 v254, v[96:99] offset:24576\n\t"
      "ds_write_b128 v254, v[100:103] offset:25600\n\t"
      "ds_write_b128 v254, v[104:107] offset:26624\n\t"
      "ds_write_b128 v254, v[108:111] offset:27648\n\t"
      "ds_write_b128 v254, v[112:115] offset:28672\n\t"
      "ds_write_b128 v254, v[116:119] offset:29696\n\t"
      "ds_write_b128 v254, v[120:123] offset:30720\n\t"
      "ds_write_b128 v254, v[124:127] offset:31744\n\t"
      "s_waitcnt lgkmcnt(0)\n\t"
      "s_barrier\n\t"
      "s_branch 9f\n\t"
      "5:\n\t"
      "s_waitcnt lgkmcnt(0) vmcnt(0)\n\t"
      "s_barrier\n\t"
      "v_add_u32 v254, %[lx], %[v16]\n\t"
      "v_lshrrev_b32 v255, 8, %[v16]\n\t"
      "v_and_b32 %[vc], 0xf0, %[v16]\n\t"
      "v_lshl_add_u32 %[vc], v255, 18, %[vc]\n\t"
      "v_add_u32 %[vc], %[cbase], %[vc]\n\t"
      "ds_read_b128 v[128:131], v254 offset:0\n\t"
      "ds_read_b128 v[132:135], v254 offset:1024\n\t"
      "ds_read_b128 v[136:139], v254 offset:2048\n\t"
      "ds_read_b128 v[140:143], v254 offset:3072\n\t"
      "ds_read_b128 v[144:147], v254 offset:4096\n\t"
      "ds_read_b128 v[148:151], v254 offset:5120\n\t"
      "ds_read_b128 v[152:155], v254 offset:6144\n\t"
      "ds_read_b128 v[156:159], v254 offset:7168\n\t"
      "ds_read_b128 v[160:163], v254 offset:8192\n\t"
      "ds_read_b128 v[164:167], v254 offset:9216\n\t"
      "ds_read_b128 v[168:171], v254 offset:10240\n\t"
      "ds_read_b128 v[172:175], v254 offset:11264\n\t"
      "ds_read_b128 v[176:179], v254 offset:12288\n\t"
      "ds_read_b128 v[180:183], v254 offset:13312\n\t"
      "ds_read_b128 v[184:187], v254 offset:14336\n\t"
      "ds_read_b128 v[188:191], v254 offset:15360\n\t"
      "s_mov_b32 s78, 0\n\t"
      "s_waitcnt lgkmcnt(8)\n\t"
      "v_add_f32 v192, v0, v128\n\t"
      "v_add_f32 v193, v4, v132\n\t"
      "v_add_f32 v194, v8, v136\n\t"
      "v_add_f32 v195, v12, v140\n\t"
      "v_add_f32 v196, v16, v144\n\t"
      "v_add_f32 v197, v20, v148\n\t"
      "v_add_f32 v198, v24, v152\n\t"
      "v_add_f32 v199, v28, v156\n\t"
      "v_add_f32 v200, v1, v129\n\t"
      "v_add_f32 v201, v5, v133\n\t"
      "v_add_f32 v202, v9, v137\n\t"
      "v_add_f32 v203, v13, v141\n\t"
      "v_add_f32 v204, v17, v145\n\t"
      "v_add_f32 v205, v21, v149\n\t"
      "v_add_f32 v206, v25, v153\n\t"
      "v_add_f32 v207, v29, v157\n\t"
      "v_add_f32 v208, v2, v130\n\t"
      "v_add_f32 v209, v6, v134\n\t"
      "v_add_f32 v210, v10, v138\n\t"
      "v_add_f32 v211, v14, v142\n\t"
      "v_add_f32 v212, v18, v146\n\t"
      "v_add_f32 v213, v22, v150\n\t"
      "v_add_f32 v214, v26, v154\n\t"
      "v_add_f32 v215, v30, v158\n\t"
      "v_add_f32 v216, v3, v131\n\t"
      "v_add_f32 v217, v7, v135\n\t"
      "v_add_f32 v218, v11, v139\n\t"
      "v_add_f32 v219, v15, v143\n\t"
      "v_add_f32 v220, v19, v147\n\t"
      "v_add_f32 v221, v23, v151\n\t"
      "v_add_f32 v222, v27, v155\n\t"
      "v_add_f32 v223, v31, v159\n\t"
      "ds_read_b128 v[128:131], v254 offset:16384\n\t"
      "ds_read_b128 v[132:135], v254 offset:17408\n\t"
      "ds_read_b128 v[136:139], v254 offset:18432\n\t"
      "ds_read_b128 v[140:143], v254 offset:19456\n\t"
      "ds_read_b128 v[144:147], v254 offset:20480\n\t"
      "ds_read_b128 v[148:151], v254 offset:21504\n\t"
      "ds_read_b128 v[152:155], v254 offset:22528\n\t"
      "ds_read_b128 v[156:159], v254 offset:23552\n\t"
      "buffer_store_dwordx4 v[192:195], %[vc], %[srdc], s78 offen offset:0 nt\n\t"
      "buffer_store_dwordx4 v[196:199], %[vc], %[srdc], s78 offen offset:256 nt\n\t"
      "s_add_u32 s78, s78, 0x10000\n\t"
      "buffer_store_dwordx4 v[200:203], %[vc], %[srdc], s78 offen offset:0 nt\n\t"
      "buffer_store_dwordx4 v[204:207], %[vc], %[srdc], s78 offen offset:256 nt\n\t"
      "s_add_u32 s78, s78, 0x10000\n\t"
      "buffer_store_dwordx4 v[208:211], %[vc], %[srdc], s78 offen offset:0 nt\n\t"
      "buffer_store_dwordx4 v[212:215], %[vc], %[srdc], s78 offen offset:256 nt\n\t"
      "s_add_u32 s78, s78, 0x10000\n\t"
      "buffer_store_dwordx4 v[216:219], %[vc], %[srdc], s78 offen offset:0 nt\n\t"
      "buffer_store_dwordx4 v[220:223], %[vc], %[srdc], s78 offen offset:256 nt\n\t"
      "s_add_u32 s78, s78, 0x10000\n\t"
      "s_add_u32 s78, s78, 786432\n\t"
      "s_nop 1\n\t"
      "s_waitcnt lgkmcnt(8)\n\t"
      "v_add_f32 v192, v32, v160\n\t"
      "v_add_f32 v193, v36, v164\n\t"
      "v_add_f32 v194, v40, v168\n\t"
      "v_add_f32 v195, v44, v172\n\t"
      "v_add_f32 v196, v48, v176\n\t"
      "v_add_f32 v197, v52, v180\n\t"
      "v_add_f32 v198, v56, v184\n\t"
      "v_add_f32 v199, v60, v188\n\t"
      "v_add_f32 v200, v33, v161\n\t"
      "v_add_f32 v201, v37, v165\n\t"
      "v_add_f32 v202, v41, v169\n\t"
      "v_add_f32 v203, v45, v173\n\t"
      "v_add_f32 v204, v49, v177\n\t"
      "v_add_f32 v205, v53, v181\n\t"
      "v_add_f32 v206, v57, v185\n\t"
      "v_add_f32 v207, v61, v189\n\t"
      "v_add_f32 v208, v34, v162\n\t"
      "v_add_f32 v209, v38, v166\n\t"
      "v_add_f32 v210, v42, v170\n\t"
      "v_add_f32 v211, v46, v174\n\t"
      "v_add_f32 v212, v50, v178\n\t"
      "v_add_f32 v213, v54, v182\n\t"
      "v_add_f32 v214, v58, v186\n\t"
      "v_add_f32 v215, v62, v190\n\t"
      "v_add_f32 v216, v35, v163\n\t"
      "v_add_f32 v217, v39, v167\n\t"
      "v_add_f32 v218, v43, v171\n\t"
      "v_add_f32 v219, v47, v175\n\t"
      "v_add_f32 v220, v51, v179\n\t"
      "v_add_f32 v221, v55, v183\n\t"
      "v_add_f32 v222, v59, v187\n\t"
      "v_add_f32 v223, v63, v191\n\t"
      "ds_read_b128 v[160:163], v254 offset:24576\n\t"
      "ds_read_b128 v[164:167], v254 offset:25600\n\t"
      "ds_read_b128 v[168:171], v254 offset:26624\n\t"
      "ds_read_b128 v[172:175], v254 offset:27648\n\t"
      "ds_read_b128 v[176:179], v254 offset:28672\n\t"
      "ds_read_b128 v[180:183], v254 offset:29696\n\t"
      "ds_read_b128 v[184:187], v254 offset:30720\n\t"
      "ds_read_b128 v[188:191], v254 offset:31744\n\t"
      "buffer_store_dwordx4 v[192:195], %[vc], %[srdc], s78 offen offset:0 nt\n\t"
      "buffer_store_dwordx4 v[196:199], %[vc], %[srdc], s78 offen offset:256 nt\n\t"
      "s_add_u32 s78, s78, 0x10000\n\t"
      "buffer_store_dwordx4 v[200:203], %[vc], %[srdc], s78 offen offset:0 nt\n\t"
      "buffer_store_dwordx4 v[204:207], %[vc], %[srdc], s78 offen offset:256 nt\n\t"
      "s_add_u32 s78, s78, 0x10000\n\t"
      "buffer_store_dwordx4 v[208:211], %[vc], %[srdc], s78 offen offset:0 nt\n\t"
      "buffer_store_dwordx4 v[212:215], %[vc], %[srdc], s78 offen offset:256 nt\n\t"
      "s_add_u32 s78, s78, 0x10000\n\t"
      "buffer_store_dwordx4 v[216:219], %[vc], %[srdc], s78 offen offset:0 nt\n\t"
      "buffer_store_dwordx4 v[220:223], %[vc], %[srdc], s78 offen offset:256 nt\n\t"
      "s_add_u32 s78, s78, 0x10000\n\t"
      "s_add_u32 s78, s78, 786432\n\t"
      "s_nop 1\n\t"
      "s_waitcnt lgkmcnt(8)\n\t"
      "v_add_f32 v192, v64, v128\n\t"
      "v_add_f32 v193, v68, v132\n\t"
      "v_add_f32 v194, v72, v136\n\t"
      "v_add_f32 v195, v76, v140\n\t"
      "v_add_f32 v196, v80, v144\n\t"
      "v_add_f32 v197, v84, v148\n\t"
      "v_add_f32 v198, v88, v152\n\t"
      "v_add_f32 v199, v92, v156\n\t"
      "v_add_f32 v200, v65, v129\n\t"
      "v_add_f32 v201, v69, v133\n\t"
      "v_add_f32 v202, v73, v137\n\t"
      "v_add_f32 v203, v77, v141\n\t"
      "v_add_f32 v204, v81, v145\n\t"
      "v_add_f32 v205, v85, v149\n\t"
      "v_add_f32 v206, v89, v153\n\t"
      "v_add_f32 v207, v93, v157\n\t"
      "v_add_f32 v208, v66, v130\n\t"
      "v_add_f32 v209, v70, v134\n\t"
      "v_add_f32 v210, v74, v138\n\t"
      "v_add_f32 v211, v78, v142\n\t"
      "v_add_f32 v212, v82, v146\n\t"
      "v_add_f32 v213, v86, v150\n\t"
      "v_add_f32 v214, v90, v154\n\t"
      "v_add_f32 v215, v94, v158\n\t"
      "v_add_f32 v216, v67, v131\n\t"
      "v_add_f32 v217, v71, v135\n\t"
      "v_add_f32 v218, v75, v139\n\t"
      "v_add_f32 v219, v79, v143\n\t"
      "v_add_f32 v220, v83, v147\n\t"
      "v_add_f32 v221, v87, v151\n\t"
      "v_add_f32 v222, v91, v155\n\t"
      "v_add_f32 v223, v95, v159\n\t"
      "buffer_store_dwordx4 v[192:195], %[vc], %[srdc], s78 offen offset:0 nt\n\t"
      "buffer_store_dwordx4 v[196:199], %[vc], %[srdc], s78 offen offset:256 nt\n\t"
      "s_add_u32 s78, s78, 0x10000\n\t"
      "buffer_store_dwordx4 v[200:203], %[vc], %[srdc], s78 offen offset:0 nt\n\t"
      "buffer_store_dwordx4 v[204:207], %[vc], %[srdc], s78 offen offset:256 nt\n\t"
      "s_add_u32 s78, s78, 0x10000\n\t"
      "buffer_store_dwordx4 v[208:211], %[vc], %[srdc], s78 offen offset:0 nt\n\t"
      "buffer_store_dwordx4 v[212:215], %[vc], %[srdc], s78 offen offset:256 nt\n\t"
      "s_add_u32 s78, s78, 0x10000\n\t"
      "buffer_store_dwordx4 v[216:219], %[vc], %[srdc], s78 offen offset:0 nt\n\t"
      "buffer_store_dwordx4 v[220:223], %[vc], %[srdc], s78 offen offset:256 nt\n\t"
      "s_add_u32 s78, s78, 0x10000\n\t"
      "s_add_u32 s78, s78, 786432\n\t"
      "s_nop 1\n\t"
      "s_waitcnt lgkmcnt(0)\n\t"
      "v_add_f32 v192, v96, v160\n\t"
      "v_add_f32 v193, v100, v164\n\t"
      "v_add_f32 v194, v104, v168\n\t"
      "v_add_f32 v195, v108, v172\n\t"
      "v_add_f32 v196, v112, v176\n\t"
      "v_add_f32 v197, v116, v180\n\t"
      "v_add_f32 v198, v120, v184\n\t"
      "v_add_f32 v199, v124, v188\n\t"
      "v_add_f32 v200, v97, v161\n\t"
      "v_add_f32 v201, v101, v165\n\t"
      "v_add_f32 v202, v105, v169\n\t"
      "v_add_f32 v203, v109, v173\n\t"
      "v_add_f32 v204, v113, v177\n\t"
      "v_add_f32 v205, v117, v181\n\t"
      "v_add_f32 v206, v121, v185\n\t"
      "v_add_f32 v207, v125, v189\n\t"
      "v_add_f32 v208, v98, v162\n\t"
      "v_add_f32 v209, v102, v166\n\t"
      "v_add_f32 v210, v106, v170\n\t"
      "v_add_f32 v211, v110, v174\n\t"
      "v_add_f32 v212, v114, v178\n\t"
      "v_add_f32 v213, v118, v182\n\t"
      "v_add_f32 v214, v122, v186\n\t"
      "v_add_f32 v215, v126, v190\n\t"
      "v_add_f32 v216, v99, v163\n\t"
      "v_add_f32 v217, v103, v167\n\t"
      "v_add_f32 v218, v107, v171\n\t"
      "v_add_f32 v219, v111, v175\n\t"
      "v_add_f32 v220, v115, v179\n\t"
      "v_add_f32 v221, v119, v183\n\t"
      "v_add_f32 v222, v123, v187\n\t"
      "v_add_f32 v223, v127, v191\n\t"
      "buffer_store_dwordx4 v[192:195], %[vc], %[srdc], s78 offen offset:0 nt\n\t"
      "buffer_store_dwordx4 v[196:199], %[vc], %[srdc], s78 offen offset:256 nt\n\t"
      "s_add_u32 s78, s78, 0x10000\n\t"
      "buffer_store_dwordx4 v[200:203], %[vc], %[srdc], s78 offen offset:0 nt\n\t"
      "buffer_store_dwordx4 v[204:207], %[vc], %[srdc], s78 offen offset:256 nt\n\t"
      "s_add_u32 s78, s78, 0x10000\n\t"
      "buffer_store_dwordx4 v[208:211], %[vc], %[srdc], s78 offen offset:0 nt\n\t"
      "buffer_store_dwordx4 v[212:215], %[vc], %[srdc], s78 offen offset:256 nt\n\t"
      "s_add_u32 s78, s78, 0x10000\n\t"
      "buffer_store_dwordx4 v[216:219], %[vc], %[srdc], s78 offen offset:0 nt\n\t"
      "buffer_store_dwordx4 v[220:223], %[vc], %[srdc], s78 offen offset:256 nt\n\t"
      "s_add_u32 s78, s78, 0x10000\n\t"
      "s_add_u32 s78, s78, 786432\n\t"
      "s_nop 1\n\t"
      "9:\n\t"
      "s_waitcnt vmcnt(0)\n\t"
    : [vp] "+v"(vp), [vc] "+v"(vc)
    : [v16] "v"(v16), [vw] "v"(vw), [tk] "v"(tk), [vx] "v"(vx), [cbase] "s"(RFL(cbase)), [pj] "s"(RFL(pj)), [srdw] "s"(srdw), [srdx] "s"(srdx), [srdc] "s"(srdc), [srds] "s"(srds), [srdd] "s"(srdd),
      [abase] "s"(RFL(abase)), [dbase] "s"(RFL(dbase)), [fdst] "s"(RFL(fdst)), [fsrc] "s"(RFL(fsrc)), [cnt] "s"(RFL(cnt)), [tmo] "s"(tmo),
      [lw] "s"(RFL(lw)), [lr] "s"(RFL(lr)), [lx] "s"(RFL(lx)), [grp] "s"(RFL(grp)), [chunk] "s"(RFL(chunk)), [send] "s"(RFL(send)), [pub] "s"(RFL(pub)), [xloc] "s"(RFL(xloc)), [dbg2] "s"(RFL(dbg2))
    : VCLOB, SCLOB, "vcc", "scc", "memory");
}

extern "C" void kernel_launch(int rank, int nranks, ncclComm_t comm,
                              hipStream_t stream,
                              void* const* d_in, const int* in_sizes, int n_in,
                              void* d_out, int out_size,
                              const int* in_dtypes, int out_dtype,
                              void* const* d_out_all,
                              void* const* d_ws_all, size_t ws_bytes) {
  (void)comm; (void)in_sizes; (void)n_in; (void)out_size; (void)in_dtypes; (void)out_dtype; (void)d_out_all;
  if (nranks != NR || (unsigned)rank >= (unsigned)NR || ws_bytes < WS_USED) {
    fprintf(stderr, "unsupported configuration\n");
    return;
  }
  a2a_gemm_kernel<<<256, 512, 0, stream>>>((const float*)d_in[0], (const float*)d_in[1], (float*)d_out, d_ws_all, rank);
  HIP_CHECK(hipGetLastError());
}
```
